```python
import numpy as np

kernel_name = "gpu_gemm_rs_m4096_k4096_n4096_e4m3e5m2_dqsilu"


ABSMAX_RTOL = 0.05

COMPUTE_DTYPE = "float32"
EPILOGUE = "dequant_silu"
IN_QUANT = None
OUT_QUANT = None
IN_FP8 = "e4m3e5m2"
IN_DTYPES = {"x": "e4m3", "w_mat": "e5m2",
             "scale_x": "float32", "scale_w": "float32"}
WORLD_SIZE = 8

def _snap_fp8(x, mant_bits, max_abs, min_normal):
    a = np.abs(x.astype(np.float64))
    _m, _e = np.frexp(a)
    q = np.where(a >= min_normal, np.ldexp(1.0, _e - 1 - mant_bits),
                 min_normal * 2.0 ** -mant_bits)
    snapped = np.minimum(np.round(a / q) * q, max_abs)
    return (np.sign(x) * snapped).astype(np.float32)

def setup_inputs():

    x = _snap_fp8(np.random.randn(4096, 4096).astype(np.float32), 3, 448.0, 2.0 ** -6)
    w_mat = _snap_fp8(np.random.randn(4096, 4096).astype(np.float32), 2, 57344.0, 2.0 ** -14)
    scale_x = (0.02 * (1.0 + np.random.rand(1))).astype(np.float32)
    scale_w = (0.02 * (1.0 + np.random.rand(1))).astype(np.float32)
    return {"x": x, "w_mat": w_mat, "scale_x": scale_x, "scale_w": scale_w}

def reference(x, w_mat, scale_x, scale_w):
    acc = (x @ w_mat).astype(np.float32)
    y = (acc * (scale_x * scale_w)).astype(np.float32)
    return (y / (1.0 + np.exp(-np.clip(y, -60.0, 60.0)))).astype(np.float32)
```

```cpp
#include <hip/hip_runtime.h>
#include <rccl/rccl.h>
#include <cstdio>

typedef unsigned u32x4 __attribute__((ext_vector_type(4)));
typedef int v8i __attribute__((ext_vector_type(8)));
typedef float f32x4 __attribute__((ext_vector_type(4)));
typedef __attribute__((address_space(1))) unsigned gu32;

#define POLL_SLEEP 16
#define NO_REL
#define NO_ACQ
#define OUT_AUX 17
constexpr int NR = 8;
constexpr int NN = 4096;
constexpr int KS = 512;
constexpr int MS = 512;

constexpr unsigned XBUF_SRC = 256u << 10;
constexpr unsigned XBUF_PAR = NR * XBUF_SRC;
constexpr unsigned XBUF = 0;
constexpr unsigned WBUF_SRC = 2u << 20;
constexpr unsigned WBUF_PAR = NR * WBUF_SRC;
constexpr unsigned WBUF = 2 * XBUF_PAR;
constexpr unsigned FLAGS = WBUF + 2 * WBUF_PAR;
constexpr unsigned WFLAG = FLAGS;
constexpr unsigned XFLAG = FLAGS + (16u << 10);
constexpr unsigned CNT = FLAGS + (32u << 10);
constexpr unsigned TMO = FLAGS + (64u << 10);
constexpr unsigned WS_USED = FLAGS + (128u << 10);

constexpr int NWP = 32;
constexpr int NXP = 64;
constexpr unsigned long long DEADLINE_TICKS = 50000000ull;

#define STAMP(i) do {} while (0)

struct Params {
  char* ws[NR];
  const unsigned char* x;
  const unsigned char* w;
  const float* sx;
  const float* sw;
  float* out;
  int rank;
};

__device__ __forceinline__ __amdgpu_buffer_rsrc_t mk_rsrc(const void* p, unsigned bytes) {
  return __builtin_amdgcn_make_buffer_rsrc(const_cast<void*>(p), 0, (int)bytes, 0x00020000);
}

__device__ __forceinline__ void transpose4(unsigned r0, unsigned r1, unsigned r2, unsigned r3,
                                           unsigned& o0, unsigned& o1, unsigned& o2, unsigned& o3) {
  const unsigned t0 = __builtin_amdgcn_perm(r1, r0, 0x05010400u);
  const unsigned t1 = __builtin_amdgcn_perm(r1, r0, 0x07030602u);
  const unsigned t2 = __builtin_amdgcn_perm(r3, r2, 0x05010400u);
  const unsigned t3 = __builtin_amdgcn_perm(r3, r2, 0x07030602u);
  o0 = __builtin_amdgcn_perm(t2, t0, 0x05040100u);
  o1 = __builtin_amdgcn_perm(t2, t0, 0x07060302u);
  o2 = __builtin_amdgcn_perm(t3, t1, 0x05040100u);
  o3 = __builtin_amdgcn_perm(t3, t1, 0x07060302u);
}

__global__ void __launch_bounds__(320, 2) gemm_rs_kernel(Params P) {
  const int wave = __builtin_amdgcn_readfirstlane(threadIdx.x >> 6);
  const int lane = threadIdx.x & 63;
  const int b = blockIdx.x;
  const int rank = P.rank;
  char* const myws = P.ws[rank];

  if (wave == 4 && b >= NWP + NXP) return;

  gu32* cntp = (gu32*)(myws + CNT) + b * 8 + wave;
  const unsigned call = __builtin_amdgcn_readfirstlane(*cntp) + 1u;
  if (lane == 0) *cntp = call;
  const unsigned par = call & 1u;
  STAMP(0);

  if (wave == 4) {
    const int c = lane & 15, g = lane >> 4;
    if (b < NWP) {
      const int v = b;
      __amdgpu_buffer_rsrc_t rw = mk_rsrc(P.w, KS * NN);
      __amdgpu_buffer_rsrc_t rs[NR];
#pragma unroll
      for (int j = 0; j < NR; ++j) rs[j] = mk_rsrc(P.ws[(rank + 1 + j + v) & 7], WS_USED);
      gu32* fbase = (gu32*)(P.ws[lane & 7] + WFLAG) + rank * 4;
      asm volatile("" : "+v"(fbase));
      const unsigned lsrc = (unsigned)(16 * g) * NN + 4 * c;
      unsigned raw[32];
      {
        const int nt = ((v >> 2) + 8 * rank) & 63, ks = v & 3;
        const unsigned so = (unsigned)(128 * ks) * NN + 64 * nt;
#pragma unroll
        for (int pl = 0; pl < 2; ++pl)
#pragma unroll
          for (int j = 0; j < 16; ++j)
            raw[pl * 16 + j] = __builtin_amdgcn_raw_buffer_load_b32(rw, lsrc + (64 * pl + j) * NN, so, 0);
      }
      for (int it = v; it < 256; it += NWP) {
        const int nt = ((it >> 2) + 8 * rank) & 63, ks = it & 3;
        u32x4 o[4][2];
#pragma unroll
        for (int pl = 0; pl < 2; ++pl)
#pragma unroll
          for (int jj = 0; jj < 4; ++jj) {
            unsigned a0, a1, a2, a3;
            transpose4(raw[pl * 16 + 4 * jj], raw[pl * 16 + 4 * jj + 1], raw[pl * 16 + 4 * jj + 2],
                       raw[pl * 16 + 4 * jj + 3], a0, a1, a2, a3);
            o[0][pl][jj] = a0; o[1][pl][jj] = a1; o[2][pl][jj] = a2; o[3][pl][jj] = a3;
          }
        {
          const int nx = min(it + NWP, 255);
          const int nt2 = ((nx >> 2) + 8 * rank) & 63, ks2 = nx & 3;
          const unsigned so = (unsigned)(128 * ks2) * NN + 64 * nt2;
#pragma unroll
          for (int pl = 0; pl < 2; ++pl)
#pragma unroll
            for (int j = 0; j < 16; ++j)
              raw[pl * 16 + j] = __builtin_amdgcn_raw_buffer_load_b32(rw, lsrc + (64 * pl + j) * NN, so, 0);
        }
        const unsigned dsto = WBUF + par * WBUF_PAR + rank * WBUF_SRC + (nt * 4 + ks) * 8192 + lane * 16;
#pragma unroll
        for (int i = 0; i < 4; ++i)
#pragma unroll
          for (int pl = 0; pl < 2; ++pl) {
#pragma unroll
            for (int j = 0; j < NR; ++j)
              __builtin_amdgcn_raw_buffer_store_b128(o[i][pl], rs[j], dsto + (i * 2 + pl) * 1024, 0, 17);
          }
        asm volatile("s_waitcnt vmcnt(0)" ::: "memory");
        if (lane < NR) __hip_atomic_store(fbase + nt * 32 + ks, call, __ATOMIC_RELAXED, __HIP_MEMORY_SCOPE_SYSTEM);
        STAMP(1 + it / NWP);
      }
    } else {
      const int u = b - NWP, q = (u >> 3) & 7, rb = u & 7;
      __amdgpu_buffer_rsrc_t rx = mk_rsrc(P.x, 4096 * KS);
      __amdgpu_buffer_rsrc_t rd = mk_rsrc(P.ws[q], WS_USED);
      const unsigned so = (unsigned)(MS * q + 64 * rb) * KS;
      const unsigned lsrc = (unsigned)c * KS + 16 * g;
      u32x4 d[32];
#pragma unroll
      for (int f = 0; f < 4; ++f)
#pragma unroll
        for (int h = 0; h < 8; ++h)
          d[f * 8 + h] = __builtin_amdgcn_raw_buffer_load_b128(rx, lsrc + 16 * f * KS + 64 * h, so, 0);
      const unsigned dsto = XBUF + par * XBUF_PAR + rank * XBUF_SRC + rb * 32768 + lane * 16;
#pragma unroll
      for (int t = 0; t < 32; ++t)
        __builtin_amdgcn_raw_buffer_store_b128(d[t], rd, dsto + t * 1024, 0, 17);
      asm volatile("s_waitcnt vmcnt(0)" ::: "memory");
      if (lane == 0)
        __hip_atomic_store((gu32*)(P.ws[q] + XFLAG) + rb * 32 + rank, call, __ATOMIC_RELAXED, __HIP_MEMORY_SCOPE_SYSTEM);
      STAMP(1);
    }
    return;
  }

  const int xc = b & 7, idx = b >> 3;
  const int nt = xc * 8 + (idx >> 2);
  const int mw = (idx & 3) * 4 + wave;
  __amdgpu_buffer_rsrc_t rl = mk_rsrc(myws, WS_USED);
  gu32* tmo = (gu32*)(myws + TMO);
  const unsigned long long t0 = __builtin_amdgcn_s_memrealtime();
  bool dead = false;

  {
    const unsigned fo = XFLAG + (mw >> 1) * 128 + (lane & 7) * 4;
    for (;;) {
      asm volatile("" ::: "memory");
      const unsigned f = __builtin_amdgcn_raw_buffer_load_b32(rl, fo, 0, 17);
      if (__all(f >= call)) break;
      if (__hip_atomic_load(tmo, __ATOMIC_RELAXED, __HIP_MEMORY_SCOPE_AGENT) == call ||
          __builtin_amdgcn_s_memrealtime() - t0 > DEADLINE_TICKS) {
        __hip_atomic_store(tmo, call, __ATOMIC_RELAXED, __HIP_MEMORY_SCOPE_AGENT);
        dead = true;
        break;
      }
      __builtin_amdgcn_s_sleep(POLL_SLEEP);
    }
  }

  STAMP(1);
  f32x4 acc[2][4];
#pragma unroll
  for (int m = 0; m < 2; ++m)
#pragma unroll
    for (int i = 0; i < 4; ++i) acc[m][i] = f32x4{0.f, 0.f, 0.f, 0.f};

  const unsigned xbase = XBUF + par * XBUF_PAR + mw * 16384;
  const unsigned wbase = WBUF + par * WBUF_PAR + nt * 32768;
  for (int s = 0; s < NR; ++s) {
    const int p = (xc - s) & 7;
    u32x4 A[2][4][2];
#pragma unroll
    for (int m = 0; m < 2; ++m)
#pragma unroll
      for (int h = 0; h < 8; ++h)
        A[m][h >> 1][h & 1] =
            __builtin_amdgcn_raw_buffer_load_b128(rl, lane * 16 + (m * 8 + h) * 1024, xbase + p * XBUF_SRC, 0);
    {
      const unsigned fo = WFLAG + nt * 128 + p * 16;
      for (;;) {
        asm volatile("" ::: "memory");
        const u32x4 f = __builtin_amdgcn_raw_buffer_load_b128(rl, fo, 0, 17);
        const unsigned mn = min(min(f[0], f[1]), min(f[2], f[3]));
        if (__builtin_amdgcn_readfirstlane(mn) >= call) break;
        if (dead) break;
        if (__hip_atomic_load(tmo, __ATOMIC_RELAXED, __HIP_MEMORY_SCOPE_AGENT) == call ||
            __builtin_amdgcn_s_memrealtime() - t0 > DEADLINE_TICKS) {
          __hip_atomic_store(tmo, call, __ATOMIC_RELAXED, __HIP_MEMORY_SCOPE_AGENT);
          dead = true;
          break;
        }
        __builtin_amdgcn_s_sleep(POLL_SLEEP);
      }
    }
    STAMP(2 + s);
    u32x4 B[4][4][2];
#pragma unroll
    for (int t = 0; t < 32; ++t)
      B[t >> 3][(t >> 1) & 3][t & 1] =
          __builtin_amdgcn_raw_buffer_load_b128(rl, lane * 16 + t * 1024, wbase + p * WBUF_SRC, 0);
    __builtin_amdgcn_sched_barrier(0);
#pragma unroll
    for (int ks = 0; ks < 4; ++ks)
#pragma unroll
      for (int i = 0; i < 4; ++i) {
        const v8i bb = {(int)B[ks][i][0][0], (int)B[ks][i][0][1], (int)B[ks][i][0][2], (int)B[ks][i][0][3],
                        (int)B[ks][i][1][0], (int)B[ks][i][1][1], (int)B[ks][i][1][2], (int)B[ks][i][1][3]};
#pragma unroll
        for (int m = 0; m < 2; ++m) {
          const v8i aa = {(int)A[m][ks][0][0], (int)A[m][ks][0][1], (int)A[m][ks][0][2], (int)A[m][ks][0][3],
                          (int)A[m][ks][1][0], (int)A[m][ks][1][1], (int)A[m][ks][1][2], (int)A[m][ks][1][3]};
          acc[m][i] = __builtin_amdgcn_mfma_scale_f32_16x16x128_f8f6f4(aa, bb, acc[m][i], 0, 1, 0, 0x7f7f7f7f, 0,
                                                                      0x7f7f7f7f);
        }
      }
  }

  STAMP(10);
  const bool bad = __hip_atomic_load(tmo, __ATOMIC_RELAXED, __HIP_MEMORY_SCOPE_AGENT) == call;
  const float scale = (*P.sx) * (*P.sw);
  const int c = lane & 15, g = lane >> 4;
  float* orow = P.out + (size_t)(32 * mw + 4 * g) * NN + 64 * nt + 4 * c;
  __amdgpu_buffer_rsrc_t ro = mk_rsrc(P.out, MS * NN * 4);
  const unsigned oo = ((32 * mw + 4 * g) * NN + 64 * nt + 4 * c) * 4;
#pragma unroll
  for (int m = 0; m < 2; ++m)
#pragma unroll
    for (int r = 0; r < 4; ++r) {
      f32x4 o;
#pragma unroll
      for (int i = 0; i < 4; ++i) {
        const float y = acc[m][i][r] * scale;
        const float yc = __builtin_amdgcn_fmed3f(y, -60.f, 60.f);
        const float e = __builtin_amdgcn_exp2f(yc * -1.4426950408889634f);
        o[i] = y * __builtin_amdgcn_rcpf(1.f + e);
        if (bad) o[i] = __builtin_nanf("");
      }
      __builtin_amdgcn_raw_buffer_store_b128(__builtin_bit_cast(u32x4, o), ro, oo + (16 * m + r) * NN * 4, 0, OUT_AUX);
    }
  STAMP(11);
}

extern "C" void kernel_launch(int rank, int nranks, ncclComm_t comm, hipStream_t stream, void* const* d_in,
                              const int* in_sizes, int n_in, void* d_out, int out_size, const int* in_dtypes,
                              int out_dtype, void* const* d_out_all, void* const* d_ws_all, size_t ws_bytes) {
  if (nranks != NR || ws_bytes < WS_USED || (unsigned)rank >= (unsigned)NR) {
    fprintf(stderr, "unexpected configuration\n");
    return;
  }
  Params P;
  for (int p = 0; p < NR; ++p) P.ws[p] = (char*)d_ws_all[p];
  P.x = (const unsigned char*)d_in[0];
  P.w = (const unsigned char*)d_in[1];
  P.sx = (const float*)d_in[2];
  P.sw = (const float*)d_in[3];
  P.out = (float*)d_out;
  P.rank = rank;
  hipLaunchKernelGGL(gemm_rs_kernel, dim3(256), dim3(320), 0, stream, P);
}
```

```cpp
#include <hip/hip_runtime.h>
#include <rccl/rccl.h>
#include <cstdio>

typedef unsigned u32x4 __attribute__((ext_vector_type(4)));
typedef unsigned u32x2 __attribute__((ext_vector_type(2)));
typedef int v8i __attribute__((ext_vector_type(8)));
typedef float f32x4 __attribute__((ext_vector_type(4)));
typedef __attribute__((address_space(1))) unsigned gu32;
typedef __attribute__((address_space(3))) unsigned lu32;
typedef __attribute__((address_space(3))) unsigned short lu16;
typedef __attribute__((address_space(3))) u32x4 lu32x4;

#define POLL_SLEEP 4
#define G_SLEEP 1
#define OUT_AUX 17
constexpr int NR = 8;
constexpr int NN = 4096;
constexpr int KS = 512;
constexpr int MS = 512;

constexpr unsigned XBUF_SRC = 256u << 10;
constexpr unsigned XBUF_PAR = NR * XBUF_SRC;
constexpr unsigned XBUF = 0;
constexpr unsigned ITEM_B = 9u << 10;
constexpr unsigned WBUF_SRC = 256u * ITEM_B;
constexpr unsigned WBUF_PAR = NR * WBUF_SRC;
constexpr unsigned WBUF = 2 * XBUF_PAR;
constexpr unsigned FLAGS = WBUF + 2 * WBUF_PAR;
constexpr unsigned WFLAG = FLAGS;
constexpr unsigned XFLAG = FLAGS + (16u << 10);
constexpr unsigned CNT = FLAGS + (32u << 10);
constexpr unsigned TMO = FLAGS + (64u << 10);
constexpr unsigned WCTR = FLAGS + (65u << 10);
constexpr unsigned WS_USED = FLAGS + (128u << 10);
constexpr unsigned OOB = 0xFFFFFF00u;
constexpr int EXC_CAP = 248;

#define NWP_ 16
constexpr int NWP = NWP_;
constexpr unsigned long long DEADLINE_TICKS = 50000000ull;

constexpr int NSLOT = 8;
constexpr int LDS_CTL = NSLOT * 8192;
constexpr int LDS_HIST = LDS_CTL + 1024;
constexpr int LDS_EXC = LDS_HIST + 16384;
constexpr int LDS_BYTES = 96 * 1024;
constexpr int NGW = 8;
constexpr int NLD = 4;

#define STAMP(i) do {} while (0)

struct Params {
  char* ws[NR];
  const unsigned char* x;
  const unsigned char* w;
  const float* sx;
  const float* sw;
  float* out;
  int rank;
};

__device__ __forceinline__ __amdgpu_buffer_rsrc_t mk_rsrc(const void* p, unsigned bytes) {
  return __builtin_amdgcn_make_buffer_rsrc(const_cast<void*>(p), 0, (int)bytes, 0x00020000);
}

__device__ __forceinline__ void transpose4(unsigned r0, unsigned r1, unsigned r2, unsigned r3,
                                           unsigned& o0, unsigned& o1, unsigned& o2, unsigned& o3) {
  const unsigned t0 = __builtin_amdgcn_perm(r1, r0, 0x05010400u);
  const unsigned t1 = __builtin_amdgcn_perm(r1, r0, 0x07030602u);
  const unsigned t2 = __builtin_amdgcn_perm(r3, r2, 0x05010400u);
  const unsigned t3 = __builtin_amdgcn_perm(r3, r2, 0x07030602u);
  o0 = __builtin_amdgcn_perm(t2, t0, 0x05040100u);
  o1 = __builtin_amdgcn_perm(t2, t0, 0x07060302u);
  o2 = __builtin_amdgcn_perm(t3, t1, 0x05040100u);
  o3 = __builtin_amdgcn_perm(t3, t1, 0x07060302u);
}

__device__ __forceinline__ bool timed_out(gu32* tmo, unsigned call, unsigned long long t0, unsigned& spins) {
  if ((++spins & 63u) != 0u) return false;
  if (__hip_atomic_load(tmo, __ATOMIC_RELAXED, __HIP_MEMORY_SCOPE_AGENT) == call ||
      __builtin_amdgcn_s_memrealtime() - t0 > DEADLINE_TICKS) {
    __hip_atomic_store(tmo, call, __ATOMIC_RELAXED, __HIP_MEMORY_SCOPE_AGENT);
    return true;
  }
  return false;
}

__device__ __forceinline__ int stream_item(int it, int rank) {
  return ((((it >> 5) + rank) & 7) * 8 + (it & 7)) * 4 + ((it >> 3) & 3);
}
__device__ __forceinline__ int src_order(int s, int xc, int rank) {
  if (s == 0) return rank;
  const int own = (xc - rank) & 7;
  const int slot = (s - 1 < own) ? s - 1 : s;
  return (xc - slot) & 7;
}

__device__ __forceinline__ unsigned predict_hi(unsigned v, unsigned tlo, unsigned thi) {
  return __builtin_amdgcn_perm(thi, tlo, (v >> 2) & 0x07070707u);
}
__device__ __forceinline__ unsigned bfi(unsigned mask, unsigned a, unsigned b) { return (a & mask) | (b & ~mask); }

__global__ void __launch_bounds__(768, 3) gemm_rs_kernel(Params P) {
  __shared__ __attribute__((aligned(16))) char smem[LDS_BYTES];
  const int wave = __builtin_amdgcn_readfirstlane(threadIdx.x >> 6);
  const int lane = threadIdx.x & 63;
  const int b = blockIdx.x;
  const int rank = P.rank;
  char* const myws = P.ws[rank];
  lu32* const ctl = (lu32*)(smem + LDS_CTL);

  const int xc = b & 7, idx = b >> 3;
  const bool service = idx < 4 || idx >= 20;
  if (service && (wave != 0 || idx >= 20 + NWP / 8)) return;

  gu32* cntp = (gu32*)(myws + CNT) + b * 16 + wave;
  const unsigned call = __builtin_amdgcn_readfirstlane(*cntp) + 1u;
  if (lane == 0) *cntp = call;
  const unsigned par = call & 1u;
  STAMP(0);

  if (service) {
    const int c = lane & 15, g = lane >> 4;
    if (idx >= 20) {
      const int v = (idx - 20) * 8 + xc;
      __amdgpu_buffer_rsrc_t rl = mk_rsrc(myws, WS_USED);
      __amdgpu_buffer_rsrc_t rs[NR - 1];
#pragma unroll
      for (int j = 0; j < NR - 1; ++j) rs[j] = mk_rsrc(P.ws[(rank + 1 + (j + v) % 7) & 7], WS_USED);
      gu32* fbase = (gu32*)(P.ws[(rank + 1 + lane) & 7] + WFLAG) + rank * 4;
      asm volatile("" : "+v"(fbase));
      gu32* ctr = (gu32*)(myws + WCTR) + par * 32;
      if (v == 0 && lane == 0) *((gu32*)(myws + WCTR) + (par ^ 1u) * 32) = 0u;
      unsigned one = 1u;
      asm volatile("" : "+v"(one));
      gu32* tmo = (gu32*)(myws + TMO);
      const unsigned long long t0 = __builtin_amdgcn_s_memrealtime();
      unsigned spins = 0;
      const unsigned lfo = WFLAG + lane * 128 + rank * 16;
      bool allset = false;
      auto wait_item = [&](int ai) {
        for (;;) {
          asm volatile("" ::: "memory");
          const unsigned f = __builtin_amdgcn_raw_buffer_load_b32(rl, WFLAG + (ai >> 2) * 128 + rank * 16 + (ai & 3) * 4, 0, 17);
          if (__builtin_amdgcn_readfirstlane(f) >= call) break;
          if (timed_out(tmo, call, t0, spins)) break;
          __builtin_amdgcn_s_sleep(1);
        }
      };
      wait_item(stream_item(v, rank));
      STAMP(2);
      const unsigned ibase = WBUF + par * WBUF_PAR + rank * WBUF_SRC + lane * 16;
      u32x4 o[9], on[9];
      int cur = v, nxt = v + NWP, prev = -1;
#pragma unroll
      for (int t = 0; t < 9; ++t)
        o[t] = __builtin_amdgcn_raw_buffer_load_b128(rl, ibase + stream_item(cur, rank) * ITEM_B + t * 1024, 0, 0);
      __builtin_amdgcn_s_waitcnt(0x0F70);
      u32x4 lf = {0u, 0u, 0u, 0u};
      while (cur < 256) {
        {
          const int ai = stream_item(min(nxt, 255), rank);
          if (!allset) {
            const unsigned fk = (ai & 2) ? ((ai & 1) ? lf[3] : lf[2]) : ((ai & 1) ? lf[1] : lf[0]);
            if (__builtin_amdgcn_readlane(fk, ai >> 2) < call) wait_item(ai);
          }
          const unsigned so = ibase + ai * ITEM_B;
#pragma unroll
          for (int t = 0; t < 9; ++t) on[t] = __builtin_amdgcn_raw_buffer_load_b128(rl, so + t * 1024, 0, 0);
          lf = __builtin_amdgcn_raw_buffer_load_b128(rl, allset ? OOB : lfo, 0, 17);
        }
        unsigned tk; unsigned long long sv;
        asm volatile("s_mov_b64 %1, exec\n\ts_mov_b64 exec, 1\n\tglobal_atomic_add %0, %2, %3, off sc0\n\ts_mov_b64 exec, %1"
                     : "=&v"(tk), "=&s"(sv) : "v"(ctr), "v"(one) : "memory");
        __builtin_amdgcn_sched_barrier(0);
        const unsigned dsto = ibase + stream_item(cur, rank) * ITEM_B;
        const unsigned m0 = __builtin_amdgcn_readlane(o[8][2], 0), m1 = __builtin_amdgcn_readlane(o[8][2], 32);
#pragma unroll
        for (int t = 0; t < 3; ++t)
#pragma unroll
          for (int j = 0; j < NR - 1; ++j) __builtin_amdgcn_raw_buffer_store_b128(o[t], rs[j], dsto + t * 1024, 0, 17);
        asm volatile("s_waitcnt vmcnt(21)" : "+v"(tk) :: "memory");
        allset = allset || __all(min(min(lf[0], lf[1]), min(lf[2], lf[3])) >= call);
        if (prev >= 0 && lane < NR - 1) {
          const int pi = stream_item(prev, rank);
          __hip_atomic_store(fbase + ((pi >> 2) * 32 + (pi & 3)), call, __ATOMIC_RELAXED, __HIP_MEMORY_SCOPE_SYSTEM);
        }
#pragma unroll
        for (int t = 4; t < 7; ++t)
#pragma unroll
          for (int j = 0; j < NR - 1; ++j) __builtin_amdgcn_raw_buffer_store_b128(o[t], rs[j], dsto + t * 1024, 0, 17);
        {
          const unsigned cntl = (lane < 32 ? m0 : m1) & 0xffffu;
          const unsigned eo = ((unsigned)(lane & 31) * 16u < 16u + 2u * cntl) ? dsto + 8 * 1024 : OOB;
#pragma unroll
          for (int j = 0; j < NR - 1; ++j) __builtin_amdgcn_raw_buffer_store_b128(o[8], rs[j], eo, 0, 17);
        }
        if (m0 >> 16) {
#pragma unroll
          for (int j = 0; j < NR - 1; ++j) __builtin_amdgcn_raw_buffer_store_b128(o[3], rs[j], dsto + 3 * 1024, 0, 17);
        }
        if (m1 >> 16) {
#pragma unroll
          for (int j = 0; j < NR - 1; ++j) __builtin_amdgcn_raw_buffer_store_b128(o[7], rs[j], dsto + 7 * 1024, 0, 17);
        }
        __builtin_amdgcn_sched_barrier(0);
#pragma unroll
        for (int t = 0; t < 9; ++t) o[t] = on[t];
        prev = cur; cur = nxt;
        nxt = 2 * NWP + (int)__builtin_amdgcn_readfirstlane(tk);
      }
      asm volatile("s_waitcnt vmcnt(0)" ::: "memory");
      if (prev >= 0 && lane < NR - 1) {
        const int pi = stream_item(prev, rank);
        __hip_atomic_store(fbase + ((pi >> 2) * 32 + (pi & 3)), call, __ATOMIC_RELAXED, __HIP_MEMORY_SCOPE_SYSTEM);
      }
      STAMP(1);
    } else {
      const int j = idx * 8 + xc;
      __amdgpu_buffer_rsrc_t rx = mk_rsrc(P.x, 4096 * KS);
      __amdgpu_buffer_rsrc_t rs[NR];
      unsigned so[NR];
#pragma unroll
      for (int t = 0; t < NR; ++t) {
        const int q = (rank + 1 + t + j) & 7;
        rs[t] = mk_rsrc(P.ws[q], WS_USED);
        so[t] = (unsigned)(MS * q + 16 * j) * KS;
      }
      gu32* fptr = (gu32*)(P.ws[lane & 7] + XFLAG) + j * 8 + rank;
      const unsigned lsrc = (unsigned)c * KS + 16 * g;
      const unsigned dsto = XBUF + par * XBUF_PAR + rank * XBUF_SRC + j * 8192 + lane * 16;
      u32x4 d[2][16];
#pragma unroll
      for (int t = 0; t < 16; ++t)
        d[0][t] = __builtin_amdgcn_raw_buffer_load_b128(rx, lsrc + 64 * (t >> 3), so[t & 7], 0);
#pragma unroll
      for (int st = 0; st < 4; ++st) {
        if (st < 3) {
#pragma unroll
          for (int t = 0; t < 16; ++t)
            d[(st + 1) & 1][t] =
                __builtin_amdgcn_raw_buffer_load_b128(rx, lsrc + 64 * (2 * st + 2 + (t >> 3)), so[t & 7], 0);
        }
        if (st == 0) asm volatile("" : "+v"(fptr));
        __builtin_amdgcn_sched_barrier(0);
#pragma unroll
        for (int t = 0; t < 16; ++t)
          __builtin_amdgcn_raw_buffer_store_b128(d[st & 1][t], rs[t & 7], dsto + (2 * st + (t >> 3)) * 1024, 0, 17);
        __builtin_amdgcn_sched_barrier(0);
      }
      asm volatile("s_waitcnt vmcnt(0)" ::: "memory");
      if (lane < NR) __hip_atomic_store(fptr, call, __ATOMIC_RELAXED, __HIP_MEMORY_SCOPE_SYSTEM);
      STAMP(1);
    }
    return;
  }

  const int gi = idx - 4;
  const int nt = xc * 8 + (gi >> 1);
  const int mh = gi & 1;
  __amdgpu_buffer_rsrc_t rl = mk_rsrc(myws, WS_USED);

  if (wave >= NGW) {
    const int ld = wave - NGW, ip = ld >> 1, h = ld & 1;
    if (ld == 0) {
      if (lane < 32) ctl[lane] = 0u;
      asm volatile("s_waitcnt lgkmcnt(0)" ::: "memory");
    }
    __builtin_amdgcn_s_barrier();
    __builtin_amdgcn_s_setprio(3);
    gu32* tmo = (gu32*)(myws + TMO);
    const unsigned long long t0 = __builtin_amdgcn_s_memrealtime();
    bool dead = false;
    unsigned spins = 0;
    bool xok = ld != 0;
    const unsigned xfo = XFLAG + mh * 512 + lane * 8;
    const int lq = (lane & 15) * 2 + ip;
    const int lp = src_order(lq >> 2, xc, rank);
    const unsigned wfo = WFLAG + nt * 128 + lp * 16 + (lq & 3) * 4;
    const unsigned wsrc = WBUF + par * WBUF_PAR + nt * 4 * ITEM_B + lane * 16;
    int done = 0;
    while (done < 16 || !xok) {
      asm volatile("" ::: "memory");
      const unsigned f = __builtin_amdgcn_raw_buffer_load_b32(rl, wfo, 0, 17);
      bool progress = false;
      if (!xok) {
        const u32x2 xf = __builtin_amdgcn_raw_buffer_load_b64(rl, xfo, 0, 17);
        if (dead || __all(min(xf[0], xf[1]) >= call)) {
          xok = true; progress = true;
          if (lane == 0) __hip_atomic_store(ctl + 24, 1u, __ATOMIC_RELAXED, __HIP_MEMORY_SCOPE_WORKGROUP);
          STAMP(1);
        }
      }
      const unsigned rel = __hip_atomic_load(ctl + 16 + (lq & (NSLOT - 1)), __ATOMIC_RELAXED, __HIP_MEMORY_SCOPE_WORKGROUP);
      const bool ok = (dead || f >= call) && rel >= (unsigned)(NGW * (lq / NSLOT));
      const unsigned long long m = (__ballot(ok) & 0xffffull) >> done;
      const int n = done + __builtin_ctzll(~m);
      for (int r = done; r < n; ++r) {
        const int q = 2 * r + ip, slot = q & (NSLOT - 1);
        const int p = __builtin_amdgcn_readlane(lp, r);
        const unsigned so = wsrc + p * WBUF_SRC + (q & 3) * ITEM_B;
        u32x4 cp[4];
#pragma unroll
        for (int t = 0; t < 4; ++t) cp[t] = __builtin_amdgcn_raw_buffer_load_b128(rl, so + (4 * h + t) * 1024, 0, 0);
        const unsigned eo = so - lane * 16 + 8 * 1024 + h * 512;
        const u32x4 hd = __builtin_amdgcn_raw_buffer_load_b128(rl, eo, 0, 0);
        unsigned en2 = __builtin_amdgcn_raw_buffer_load_b32(rl, eo + 16 + lane * 4, 0, 0);
        const unsigned tlo = __builtin_amdgcn_readfirstlane(hd[0]), thi = __builtin_amdgcn_readfirstlane(hd[1]);
        const unsigned meta = __builtin_amdgcn_readfirstlane(hd[2]);
        lu32x4* dst = (lu32x4*)(smem + slot * 8192 + h * 1024 + lane * 16);
        if ((meta >> 16) == 0u) {
#pragma unroll
          for (int d = 0; d < 4; ++d) {
            const unsigned pq = cp[0][d], qq = cp[1][d], rq = cp[2][d];
            const unsigned dd = ((pq >> 5) & 0x03030303u) | ((qq >> 3) & 0x0C0C0C0Cu) | ((rq << 1) & 0x80808080u) |
                                ((rq >> 1) & 0x10101010u);
            cp[0][d] = (pq & 0x9F9F9F9Fu) | predict_hi(pq, tlo, thi);
            cp[1][d] = (qq & 0x9F9F9F9Fu) | predict_hi(qq, tlo, thi);
            cp[2][d] = (rq & 0x9F9F9F9Fu) | predict_hi(rq, tlo, thi);
            cp[3][d] = dd | predict_hi(dd, tlo, thi);
          }
        }
#pragma unroll
        for (int t = 0; t < 4; ++t) dst[t * 128] = cp[t];
        {
          const int cnt = (int)(meta & 0xffffu);
          for (int e0 = 0; e0 < cnt; e0 += 128) {
            if (e0) en2 = __builtin_amdgcn_raw_buffer_load_b32(rl, eo + 16 + e0 * 2 + lane * 4, 0, 0);
#pragma unroll
            for (int k = 0; k < 2; ++k) {
              const unsigned en = (en2 >> (16 * k)) & 0xffffu;
              if (e0 + 2 * lane + k < cnt)
                __hip_atomic_fetch_xor((lu32*)smem + (slot * 2048 + ((en >> 4) & 0x7ffu)),
                                       (en & 3u) << (((en >> 2) & 3u) * 8u + 5u), __ATOMIC_RELAXED,
                                       __HIP_MEMORY_SCOPE_WORKGROUP);
            }
          }
        }
        asm volatile("s_waitcnt lgkmcnt(0)" ::: "memory");
        if (lane == 0)
          __hip_atomic_store(ctl + slot * 2 + h, (unsigned)(q + 1), __ATOMIC_RELAXED, __HIP_MEMORY_SCOPE_WORKGROUP);
        progress = true;
      }
      done = n;
      if (!progress) {
        if (!dead && timed_out(tmo, call, t0, spins)) {
          dead = true;
          if (lane == 0) __hip_atomic_store(ctl + 25, 1u, __ATOMIC_RELAXED, __HIP_MEMORY_SCOPE_WORKGROUP);
          asm volatile("s_waitcnt lgkmcnt(0)" ::: "memory");
        } else {
          __builtin_amdgcn_s_sleep(POLL_SLEEP);
        }
      }
    }
    return;
  }

  const int mw = mh * NGW + wave;
  const float scale = (*P.sx) * (*P.sw);
  if (wave < 4) {
    const int it = nt * 4 + mh * 2 + (wave >> 1), h = wave & 1;
    __amdgpu_buffer_rsrc_t rw = mk_rsrc(P.w, KS * NN);
    unsigned raw[16];
    {
      const unsigned lsrc = (unsigned)(16 * (lane >> 4)) * NN + 4 * (lane & 15);
      const unsigned so = (unsigned)(128 * (it & 3) + 64 * h) * NN + 64 * (it >> 2);
#pragma unroll
      for (int j = 0; j < 16; ++j) raw[j] = __builtin_amdgcn_raw_buffer_load_b32(rw, lsrc + j * NN, so, 0);
    }
    lu32* hist = (lu32*)(smem + LDS_HIST + wave * 4096);
    lu16* exl = (lu16*)(smem + LDS_EXC + wave * 512);
#pragma unroll
    for (int t = 0; t < 4; ++t) ((lu32x4*)hist)[t * 64 + lane] = u32x4{0u, 0u, 0u, 0u};
    __builtin_amdgcn_s_barrier();
#pragma unroll
    for (int j = 0; j < 16; ++j)
#pragma unroll
      for (int k = 0; k < 4; ++k)
        __hip_atomic_fetch_add(hist + (((raw[j] >> (8 * k + 2)) & 31u) * 32u + (lane & 31)), 1u, __ATOMIC_RELAXED,
                               __HIP_MEMORY_SCOPE_WORKGROUP);
    u32x4 o[4];
#pragma unroll
    for (int jj = 0; jj < 4; ++jj) {
      unsigned a0, a1, a2, a3;
      transpose4(raw[4 * jj], raw[4 * jj + 1], raw[4 * jj + 2], raw[4 * jj + 3], a0, a1, a2, a3);
      o[0][jj] = a0; o[1][jj] = a1; o[2][jj] = a2; o[3][jj] = a3;
    }
    STAMP(12);
    unsigned tlo, thi;
    {
      const int e = ((lane >> 1) & 3) * 8 + (lane >> 3);
      unsigned tot = 0;
#pragma unroll
      for (int t = 0; t < 4; ++t) {
        const u32x4 hv = ((lu32x4*)hist)[e * 8 + (lane & 1) * 4 + t];
        tot += hv[0] + hv[1] + hv[2] + hv[3];
      }
      tot += (unsigned)__builtin_amdgcn_update_dpp(0, (int)tot, 0xB1, 0xf, 0xf, false);
      unsigned key = (tot << 2) | (unsigned)((lane >> 1) & 3);
      key = max(key, (unsigned)__builtin_amdgcn_update_dpp(0, (int)key, 0x4E, 0xf, 0xf, false));
      key = max(key, (unsigned)__builtin_amdgcn_update_dpp(0, (int)key, 0x141, 0xf, 0xf, false));
      const unsigned hb = (key & 3u) << 5;
      tlo = __builtin_amdgcn_readlane(hb, 0) | (__builtin_amdgcn_readlane(hb, 8) << 8) |
            (__builtin_amdgcn_readlane(hb, 16) << 16) | (__builtin_amdgcn_readlane(hb, 24) << 24);
      thi = __builtin_amdgcn_readlane(hb, 32) | (__builtin_amdgcn_readlane(hb, 40) << 8) |
            (__builtin_amdgcn_readlane(hb, 48) << 16) | (__builtin_amdgcn_readlane(hb, 56) << 24);
    }
    unsigned xm[4];
    unsigned mycnt = 0;
#pragma unroll
    for (int i = 0; i < 4; ++i) {
      unsigned acc2 = 0;
#pragma unroll
      for (int d = 0; d < 4; ++d) {
        const unsigned df = ((o[i][d] ^ predict_hi(o[i][d], tlo, thi)) >> 5) & 0x03030303u;
        acc2 |= ((df * 0x01041040u) >> 24) << (8 * d);
      }
      xm[i] = acc2;
      mycnt += (unsigned)__builtin_popcount((acc2 | (acc2 >> 1)) & 0x55555555u);
    }
    int at = 0, cnt = 0;
#pragma unroll
    for (int bit = 0; bit < 7; ++bit) {
      const unsigned long long bal = __ballot((mycnt >> bit) & 1u);
      at += (int)__builtin_amdgcn_mbcnt_hi((unsigned)(bal >> 32), __builtin_amdgcn_mbcnt_lo((unsigned)bal, 0u)) << bit;
      cnt += __builtin_popcountll(bal) << bit;
    }
    if (cnt <= EXC_CAP) {
#pragma unroll
      for (int i = 0; i < 4; ++i) {
        unsigned v = xm[i];
        while (v != 0u) {
          const unsigned f2 = (unsigned)__builtin_ctz(v) & ~1u;
          const unsigned en = ((unsigned)((2 * i + h) * 1024 + lane * 16) + (f2 >> 1)) * 4u + ((v >> f2) & 3u);
          exl[8 + at] = (unsigned short)en;
          ++at;
          v &= ~(3u << f2);
        }
      }
    }
    const bool rawm = cnt > EXC_CAP;
    const unsigned dsto = WBUF + par * WBUF_PAR + rank * WBUF_SRC + it * ITEM_B + lane * 16;
    if (!rawm) {
#pragma unroll
      for (int d = 0; d < 4; ++d) {
        const unsigned dd = o[3][d];
        o[0][d] = bfi(0x60606060u, dd << 5, o[0][d]);
        o[1][d] = bfi(0x60606060u, dd << 3, o[1][d]);
        o[2][d] = bfi(0x20202020u, dd << 1, bfi(0x40404040u, dd >> 1, o[2][d]));
      }
    }
#pragma unroll
    for (int t = 0; t < 4; ++t) __builtin_amdgcn_raw_buffer_store_b128(o[t], rl, dsto + (4 * h + t) * 1024, 0, 17);
    if (lane == 0) *(lu32x4*)exl = u32x4{tlo, thi, rawm ? 0x10000u : (unsigned)cnt, 0u};
    asm volatile("s_waitcnt lgkmcnt(0)" ::: "memory");
    {
      const u32x4 ev = ((lu32x4*)exl)[lane & 31];
      __builtin_amdgcn_raw_buffer_store_b128(ev, rl, lane < 32 ? dsto + 8 * 1024 + h * 512 : OOB, 0, 17);
    }
    asm volatile("s_waitcnt vmcnt(0)" ::: "memory");
    if (lane == 0) {
      if (__hip_atomic_fetch_add(ctl + 26 + (wave >> 1), 1u, __ATOMIC_RELAXED, __HIP_MEMORY_SCOPE_WORKGROUP) == 1u)
        __hip_atomic_store((gu32*)(myws + WFLAG) + (it >> 2) * 32 + rank * 4 + (it & 3), call, __ATOMIC_RELAXED,
                           __HIP_MEMORY_SCOPE_SYSTEM);
    }
    STAMP(13);
  } else {
    __builtin_amdgcn_s_barrier();
  }
  while (__hip_atomic_load(ctl + 24, __ATOMIC_RELAXED, __HIP_MEMORY_SCOPE_WORKGROUP) == 0u) __builtin_amdgcn_s_sleep(4);
  asm volatile("" ::: "memory");
  STAMP(1);
  f32x4 acc[2][4];
#pragma unroll
  for (int m = 0; m < 2; ++m)
#pragma unroll
    for (int i = 0; i < 4; ++i) acc[m][i] = f32x4{0.f, 0.f, 0.f, 0.f};

  const unsigned xbase = XBUF + par * XBUF_PAR + mw * 16384;
#pragma unroll 1
  for (int s = 0; s < NR; ++s) {
    const int p = src_order(s, xc, rank);
    u32x4 A[2][4][2];
#pragma unroll
    for (int m = 0; m < 2; ++m)
#pragma unroll
      for (int h = 0; h < 8; ++h)
        A[m][h >> 1][h & 1] =
            __builtin_amdgcn_raw_buffer_load_b128(rl, lane * 16 + (m * 8 + h) * 1024, xbase + p * XBUF_SRC, 0);
#pragma unroll
    for (int ks = 0; ks < 4; ++ks) {
      const int q = s * 4 + ks, slot = q & (NSLOT - 1);
      for (;;) {
        const unsigned f0 = __hip_atomic_load(ctl + slot * 2, __ATOMIC_RELAXED, __HIP_MEMORY_SCOPE_WORKGROUP);
        const unsigned f1 = __hip_atomic_load(ctl + slot * 2 + 1, __ATOMIC_RELAXED, __HIP_MEMORY_SCOPE_WORKGROUP);
        if (min(f0, f1) >= (unsigned)(q + 1)) break;
        __builtin_amdgcn_s_sleep(G_SLEEP);
      }
      asm volatile("" ::: "memory");
      u32x4 B[8];
      const lu32x4* bp = (const lu32x4*)(smem + slot * 8192 + lane * 16);
#pragma unroll
      for (int t = 0; t < 8; ++t) B[t] = bp[t * 64];
      asm volatile("s_waitcnt lgkmcnt(0)" ::: "memory");
      if (lane == 0) __hip_atomic_fetch_add(ctl + 16 + slot, 1u, __ATOMIC_RELAXED, __HIP_MEMORY_SCOPE_WORKGROUP);
#pragma unroll
      for (int i = 0; i < 4; ++i) {
        const v8i bb = {(int)B[2 * i][0], (int)B[2 * i][1], (int)B[2 * i][2], (int)B[2 * i][3],
                        (int)B[2 * i + 1][0], (int)B[2 * i + 1][1], (int)B[2 * i + 1][2], (int)B[2 * i + 1][3]};
#pragma unroll
        for (int m = 0; m < 2; ++m) {
          const v8i aa = {(int)A[m][ks][0][0], (int)A[m][ks][0][1], (int)A[m][ks][0][2], (int)A[m][ks][0][3],
                          (int)A[m][ks][1][0], (int)A[m][ks][1][1], (int)A[m][ks][1][2], (int)A[m][ks][1][3]};
          acc[m][i] = __builtin_amdgcn_mfma_scale_f32_16x16x128_f8f6f4(aa, bb, acc[m][i], 0, 1, 0, 0x7f7f7f7f, 0,
                                                                      0x7f7f7f7f);
        }
      }
      asm volatile("" : "+v"(acc[0][0]), "+v"(acc[0][1]), "+v"(acc[0][2]), "+v"(acc[0][3]), "+v"(acc[1][0]),
                   "+v"(acc[1][1]), "+v"(acc[1][2]), "+v"(acc[1][3]));
    }
    STAMP(2 + s);
  }

  STAMP(10);
  const bool bad = __builtin_amdgcn_readfirstlane(
                       __hip_atomic_load(ctl + 25, __ATOMIC_RELAXED, __HIP_MEMORY_SCOPE_WORKGROUP)) != 0u;
  const int c = lane & 15, g = lane >> 4;
  __amdgpu_buffer_rsrc_t ro = mk_rsrc(P.out, MS * NN * 4);
  const unsigned oo = ((32 * mw + 4 * g) * NN + 64 * nt + 4 * c) * 4;
  const float sc = bad ? __builtin_nanf("") : scale;
#pragma unroll
  for (int m = 0; m < 2; ++m)
#pragma unroll
    for (int r = 0; r < 4; ++r) {
      f32x4 o;
#pragma unroll
      for (int i = 0; i < 4; ++i) {
        const float y = acc[m][i][r] * sc;
        const float yc = __builtin_amdgcn_fmed3f(y, -60.f, 60.f);
        const float e = __builtin_amdgcn_exp2f(yc * -1.4426950408889634f);
        o[i] = y * __builtin_amdgcn_rcpf(1.f + e);
      }
      __builtin_amdgcn_raw_buffer_store_b128(__builtin_bit_cast(u32x4, o), ro, oo + (16 * m + r) * NN * 4, 0, OUT_AUX);
    }
  STAMP(11);
}

extern "C" void kernel_launch(int rank, int nranks, ncclComm_t comm, hipStream_t stream, void* const* d_in,
                              const int* in_sizes, int n_in, void* d_out, int out_size, const int* in_dtypes,
                              int out_dtype, void* const* d_out_all, void* const* d_ws_all, size_t ws_bytes) {
  if (nranks != NR || ws_bytes < WS_USED || (unsigned)rank >= (unsigned)NR) {
    fprintf(stderr, "unexpected configuration\n");
    return;
  }
  Params P;
  for (int p = 0; p < NR; ++p) P.ws[p] = (char*)d_ws_all[p];
  P.x = (const unsigned char*)d_in[0];
  P.w = (const unsigned char*)d_in[1];
  P.sx = (const float*)d_in[2];
  P.sw = (const float*)d_in[3];
  P.out = (float*)d_out;
  P.rank = rank;
  hipLaunchKernelGGL(gemm_rs_kernel, dim3(256), dim3(768), 0, stream, P);
}
```

```cpp
#include <hip/hip_runtime.h>
#include <rccl/rccl.h>
#include <cstdio>

typedef unsigned u32x4 __attribute__((ext_vector_type(4)));
typedef unsigned u32x2 __attribute__((ext_vector_type(2)));
typedef int v8i __attribute__((ext_vector_type(8)));
typedef float f32x4 __attribute__((ext_vector_type(4)));
typedef __attribute__((address_space(1))) unsigned gu32;
typedef __attribute__((address_space(3))) unsigned lu32;
typedef __attribute__((address_space(3))) unsigned short lu16;
typedef __attribute__((address_space(3))) u32x4 lu32x4;

#define POLL_SLEEP 4
#define SCRUB_SLEEP 8
#define EAGER_R 14
#define G_SLEEP 1
#define OUT_AUX 17
constexpr int NR = 8;
constexpr int NN = 4096;
constexpr int KS = 512;
constexpr int MS = 512;

constexpr unsigned XBUF_SRC = 256u << 10;
constexpr unsigned XBUF_PAR = NR * XBUF_SRC;
constexpr unsigned XBUF = 0;
constexpr unsigned ITEM_B = 9u << 10;
constexpr unsigned WBUF_SRC = 256u * ITEM_B;
constexpr unsigned WBUF_PAR = NR * WBUF_SRC;
constexpr unsigned WBUF = 2 * XBUF_PAR;
constexpr unsigned FLAGS = WBUF + 2 * WBUF_PAR;
constexpr unsigned WFLAG = FLAGS;
constexpr int FL2 = 2048;
constexpr unsigned XFLAG = FLAGS + (16u << 10);
constexpr unsigned CNT = FLAGS + (32u << 10);
constexpr unsigned TMO = FLAGS + (64u << 10);
constexpr unsigned WCTR = FLAGS + (65u << 10);
constexpr unsigned WS_USED = FLAGS + (128u << 10);
constexpr unsigned OOB = 0xFFFFFF00u;
constexpr int EXC_CAP = 248;
constexpr unsigned SCTR = FLAGS + (66u << 10);
constexpr unsigned WIRE_K = 0xA5C3E187u;
constexpr int NSCRUB = 80;
constexpr unsigned M_RAW = 1u << 16, M_FLAG = 1u << 17, M_VALID = 1u << 31;

#define NWP_ 16
constexpr int NWP = NWP_;
static_assert(NWP == 16, "role map assumes 16 w pushers and 80 scrubbers");
constexpr unsigned long long DEADLINE_TICKS = 50000000ull;

constexpr int NSLOT = 8;
constexpr int LDS_CTL = NSLOT * 8192;
constexpr int LDS_HIST = LDS_CTL + 1024;
constexpr int LDS_EXC = LDS_HIST + 16384;
constexpr int LDS_BYTES = 96 * 1024;
constexpr int NGW = 8;
constexpr int NLD = 4;

#define STAMP(i) do {} while (0)

struct Params {
  char* ws[NR];
  char* myws;
  const unsigned char* x;
  const unsigned char* w;
  const float* sx;
  const float* sw;
  float* out;
  int rank;
};

__device__ __forceinline__ __amdgpu_buffer_rsrc_t mk_rsrc(const void* p, unsigned bytes) {
  return __builtin_amdgcn_make_buffer_rsrc(const_cast<void*>(p), 0, (int)bytes, 0x00020000);
}

__device__ __forceinline__ void transpose4(unsigned r0, unsigned r1, unsigned r2, unsigned r3,
                                           unsigned& o0, unsigned& o1, unsigned& o2, unsigned& o3) {
  const unsigned t0 = __builtin_amdgcn_perm(r1, r0, 0x05010400u);
  const unsigned t1 = __builtin_amdgcn_perm(r1, r0, 0x07030602u);
  const unsigned t2 = __builtin_amdgcn_perm(r3, r2, 0x05010400u);
  const unsigned t3 = __builtin_amdgcn_perm(r3, r2, 0x07030602u);
  o0 = __builtin_amdgcn_perm(t2, t0, 0x05040100u);
  o1 = __builtin_amdgcn_perm(t2, t0, 0x07060302u);
  o2 = __builtin_amdgcn_perm(t3, t1, 0x05040100u);
  o3 = __builtin_amdgcn_perm(t3, t1, 0x07060302u);
}

__device__ __forceinline__ bool timed_out(gu32* tmo, unsigned call, unsigned long long t0, unsigned& spins) {
  if ((++spins & 63u) != 0u) return false;
  if (__hip_atomic_load(tmo, __ATOMIC_RELAXED, __HIP_MEMORY_SCOPE_AGENT) == call ||
      __builtin_amdgcn_s_memrealtime() - t0 > DEADLINE_TICKS) {
    __hip_atomic_store(tmo, call, __ATOMIC_RELAXED, __HIP_MEMORY_SCOPE_AGENT);
    return true;
  }
  return false;
}

__device__ __forceinline__ int stream_item(int it, int rank) {
  return ((((it >> 5) + rank) & 7) * 8 + (it & 7)) * 4 + ((it >> 3) & 3);
}
__device__ __forceinline__ int src_order(int s, int xc, int rank) {
  if (s == 0) return rank;
  const int own = (xc - rank) & 7;
  const int slot = (s - 1 < own) ? s - 1 : s;
  return (xc - slot) & 7;
}

__device__ __forceinline__ unsigned predict_hi(unsigned v, unsigned tlo, unsigned thi) {
  return __builtin_amdgcn_perm(thi, tlo, (v >> 2) & 0x07070707u);
}
__device__ __forceinline__ unsigned bfi(unsigned mask, unsigned a, unsigned b) { return (a & mask) | (b & ~mask); }

__global__ void __launch_bounds__(768, 3) gemm_rs_kernel(Params P) {
  __shared__ __attribute__((aligned(16))) char smem[LDS_BYTES];
  const int wave = __builtin_amdgcn_readfirstlane(threadIdx.x >> 6);
  const int lane = threadIdx.x & 63;
  const int b = blockIdx.x;
  const int rank = P.rank;
  char* const myws = P.myws;
  lu32* const ctl = (lu32*)(smem + LDS_CTL);

  const int xc = b & 7, idx = b >> 3;
  const bool service = idx < 4 || idx >= 20;
  if (service && wave != 0) return;

  STAMP(0);
  __amdgpu_buffer_rsrc_t rl = mk_rsrc(myws, WS_USED);
  const unsigned cnt_off = CNT + (b * 16 + wave) * 4;
  const unsigned cnt_prev = __builtin_amdgcn_raw_buffer_load_b32(rl, cnt_off, 0, 0);
#define GET_CALL()                                                                        \
  const unsigned call = __builtin_amdgcn_readfirstlane(cnt_prev) + 1u;                    \
  __builtin_amdgcn_raw_buffer_store_b32(call, rl, lane == 0 ? cnt_off : OOB, 0, 0);       \
  const unsigned par = call & 1u

  if (service) {
    const int c = lane & 15, g = lane >> 4;
    if (idx >= 22) {
      GET_CALL();
      const int sidx = (idx - 22) * 8 + xc;
      __builtin_amdgcn_s_sleep(127); __builtin_amdgcn_s_sleep(127);
      const u32x4 z = {0u, 0u, 0u, 0u};
      for (int u = sidx; u < 7 * 256; u += NSCRUB) {
        const unsigned zo = WBUF + (par ^ 1u) * WBUF_PAR + ((rank + 1 + (u >> 8)) & 7) * WBUF_SRC + (u & 255) * ITEM_B + lane * 16;
#pragma unroll
        for (int t = 0; t < 9; ++t) __builtin_amdgcn_raw_buffer_store_b128(z, rl, zo + t * 1024, 0, 17);
        asm volatile("s_waitcnt vmcnt(9)" ::: "memory");
        __builtin_amdgcn_s_sleep(SCRUB_SLEEP);
      }
      asm volatile("s_waitcnt vmcnt(0)" ::: "memory");
      if (lane == 0) __hip_atomic_fetch_add((gu32*)(myws + SCTR), 1u, __ATOMIC_RELAXED, __HIP_MEMORY_SCOPE_AGENT);
      STAMP(1);
    } else if (idx >= 20) {
      const int v = (idx - 20) * 8 + xc;
      GET_CALL();
      __amdgpu_buffer_rsrc_t rs[NR - 1];
#pragma unroll
      for (int j = 0; j < NR - 1; ++j) rs[j] = mk_rsrc(P.ws[(rank + 1 + (j + v) % 7) & 7], WS_USED);
      gu32* fbase = (gu32*)(P.ws[(rank + 1 + lane) & 7] + WFLAG) + rank * 4;
      asm volatile("" : "+v"(fbase));
      gu32* ctr = (gu32*)(myws + WCTR) + par * 32;
      if (v == 0 && lane == 0) *((gu32*)(myws + WCTR) + (par ^ 1u) * 32) = 0u;
      unsigned one = 1u;
      asm volatile("" : "+v"(one));
      gu32* tmo = (gu32*)(myws + TMO);
      const unsigned long long t0 = __builtin_amdgcn_s_memrealtime();
      unsigned spins = 0;
      const unsigned lfo = WFLAG + lane * 128 + rank * 16;
      bool allset = false;
      auto wait_item = [&](int ai) {
        for (;;) {
          asm volatile("" ::: "memory");
          const unsigned f = __builtin_amdgcn_raw_buffer_load_b32(rl, WFLAG + (ai >> 2) * 128 + rank * 16 + (ai & 3) * 4, 0, 17);
          if (__builtin_amdgcn_readfirstlane(f) >= call) break;
          if (timed_out(tmo, call, t0, spins)) break;
          __builtin_amdgcn_s_sleep(1);
        }
      };
      constexpr int NU = 512;
      const unsigned ibase = WBUF + par * WBUF_PAR + rank * WBUF_SRC;
      auto load_unit = [&](int u, u32x4 (&d)[5]) {
        const int ai = stream_item(min(u, NU - 1) >> 1, rank), hh = u & 1;
        if (!allset) {
          asm volatile("" ::: "memory");
          const u32x4 lf = __builtin_amdgcn_raw_buffer_load_b128(rl, lfo, 0, 17);
          allset = __all(min(min(lf[0], lf[1]), min(lf[2], lf[3])) >= call);
          if (!allset) wait_item(ai);
        }
        const unsigned so = ibase + ai * ITEM_B + lane * 16;
#pragma unroll
        for (int t = 0; t < 4; ++t) d[t] = __builtin_amdgcn_raw_buffer_load_b128(rl, so + (4 * hh + t) * 1024, 0, 0);
        d[4] = __builtin_amdgcn_raw_buffer_load_b128(rl, lane < 32 ? so + 8 * 1024 + hh * 512 : OOB, 0, 0);
      };
      int cur = v, n1 = v + NWP, n2 = v + 2 * NWP;
      auto step = [&](u32x4 (&oc)[5], u32x4 (&on)[5], u32x4 (&ol)[5], unsigned& tp, unsigned& tn) {
        load_unit(n2, ol);
        unsigned long long sv;
        asm volatile("s_mov_b64 %1, exec\n\ts_mov_b64 exec, 1\n\tglobal_atomic_add %0, %2, %3, off sc0\n\ts_mov_b64 exec, %1"
                     : "=&v"(tn), "=&s"(sv) : "v"(ctr), "v"(one) : "memory");
        __builtin_amdgcn_sched_barrier(0);
        const int ai = stream_item(cur >> 1, rank), hh = cur & 1;
        const unsigned dsto = ibase + ai * ITEM_B + hh * 4096 + lane * 16;
        const unsigned m0 = __builtin_amdgcn_readlane(oc[4][2], 0);
#pragma unroll
        for (int t = 0; t < 3; ++t)
#pragma unroll
          for (int j = 0; j < NR - 1; ++j) __builtin_amdgcn_raw_buffer_store_b128(oc[t], rs[j], dsto + t * 1024, 0, 17);
        if (m0 & M_RAW) {
#pragma unroll
          for (int j = 0; j < NR - 1; ++j) __builtin_amdgcn_raw_buffer_store_b128(oc[3], rs[j], dsto + 3 * 1024, 0, 17);
        }
        if (cur == 384) {
          while (__hip_atomic_load((gu32*)(myws + SCTR), __ATOMIC_RELAXED, __HIP_MEMORY_SCOPE_AGENT) < NSCRUB * call) {
            if (timed_out(tmo, call, t0, spins)) break;
            __builtin_amdgcn_s_sleep(2);
          }
        }
        {
          const unsigned eo = ((unsigned)lane * 16u < 16u + 2u * (m0 & 0xffffu))
                                  ? ibase + ai * ITEM_B + 8 * 1024 + hh * 512 + lane * 16 : OOB;
#pragma unroll
          for (int j = 0; j < NR - 1; ++j) __builtin_amdgcn_raw_buffer_store_b128(oc[4], rs[j], eo, 0, 17);
        }
        if (m0 & M_FLAG) {
          asm volatile("s_waitcnt vmcnt(0)" ::: "memory");
          if (lane < NR - 1)
            __hip_atomic_store(fbase + (FL2 * hh + (ai >> 2) * 32 + (ai & 3)), call, __ATOMIC_RELAXED, __HIP_MEMORY_SCOPE_SYSTEM);
        }
        asm volatile("s_waitcnt vmcnt(62)"
                     : "+v"(tp), "+v"(on[0]), "+v"(on[1]), "+v"(on[2]), "+v"(on[3]), "+v"(on[4]) :: "memory");
        __builtin_amdgcn_sched_barrier(0);
        cur = n1; n1 = n2;
        n2 = 4 * NWP + (int)__builtin_amdgcn_readfirstlane(tp);
      };
      u32x4 oa[5], ob[5], oc3[5];
      unsigned ta = (unsigned)(v - NWP), tb, tc;
      load_unit(cur, oa);
      STAMP(2);
      load_unit(n1, ob);
      __builtin_amdgcn_s_waitcnt(0x0F70);
      for (;;) {
        step(oa, ob, oc3, ta, tb); if (cur >= NU) break;
        step(ob, oc3, oa, tb, tc); if (cur >= NU) break;
        step(oc3, oa, ob, tc, ta); if (cur >= NU) break;
      }
      STAMP(3);
      STAMP(1);
    } else {
      const int j = idx * 8 + xc;
      __amdgpu_buffer_rsrc_t rx = mk_rsrc(P.x, 4096 * KS);
      __amdgpu_buffer_rsrc_t rs[NR];
      unsigned so[NR];
#pragma unroll
      for (int t = 0; t < NR; ++t) {
        const int q = (rank + 1 + t + j) & 7;
        rs[t] = mk_rsrc(P.ws[q], WS_USED);
        so[t] = (unsigned)(MS * q + 16 * j) * KS;
      }
      gu32* fptr = (gu32*)(P.ws[lane & 7] + XFLAG) + j * 8 + rank;
      const unsigned lsrc = (unsigned)c * KS + 16 * g;
      u32x4 d[2][16];
#pragma unroll
      for (int t = 0; t < 16; ++t)
        d[0][t] = __builtin_amdgcn_raw_buffer_load_b128(rx, lsrc + 64 * (t >> 3), so[t & 7], 0);
#pragma unroll
      for (int t = 0; t < 16; ++t)
        d[1][t] = __builtin_amdgcn_raw_buffer_load_b128(rx, lsrc + 64 * (2 + (t >> 3)), so[t & 7], 0);
      __builtin_amdgcn_sched_barrier(0);
      GET_CALL();
      const unsigned dsto = XBUF + par * XBUF_PAR + rank * XBUF_SRC + j * 8192 + lane * 16;
#pragma unroll
      for (int st = 0; st < 4; ++st) {
        if (st >= 1 && st < 3) {
#pragma unroll
          for (int t = 0; t < 16; ++t)
            d[(st + 1) & 1][t] =
                __builtin_amdgcn_raw_buffer_load_b128(rx, lsrc + 64 * (2 * st + 2 + (t >> 3)), so[t & 7], 0);
        }
        if (st == 0) asm volatile("" : "+v"(fptr));
        __builtin_amdgcn_sched_barrier(0);
#pragma unroll
        for (int t = 0; t < 16; ++t)
          __builtin_amdgcn_raw_buffer_store_b128(d[st & 1][t], rs[t & 7], dsto + (2 * st + (t >> 3)) * 1024, 0, 17);
        __builtin_amdgcn_sched_barrier(0);
      }
      asm volatile("s_waitcnt vmcnt(0)" ::: "memory");
      if (lane < NR) __hip_atomic_store(fptr, call, __ATOMIC_RELAXED, __HIP_MEMORY_SCOPE_SYSTEM);
      STAMP(1);
    }
    return;
  }

  const int gi = idx - 4;
  const int nt = xc * 8 + (gi >> 1);
  const int mh = gi & 1;

  if (wave >= NGW) {
    const int ld = wave - NGW, ip = ld >> 1, h = ld & 1;
    GET_CALL();
    if (ld == 0) {
      if (lane < 32) ctl[lane] = 0u;
      asm volatile("s_waitcnt lgkmcnt(0)" ::: "memory");
    }
    __builtin_amdgcn_s_barrier();
    __builtin_amdgcn_s_setprio(3);
    gu32* tmo = (gu32*)(myws + TMO);
    const unsigned long long t0 = __builtin_amdgcn_s_memrealtime();
    bool dead = false;
    unsigned spins = 0;
    const unsigned wsrc = WBUF + par * WBUF_PAR + nt * 4 * ITEM_B;
#pragma unroll 1
    for (int r = 0; r < 16; ++r) {
      const int q = 2 * r + ip, slot = q & (NSLOT - 1);
      const int p = src_order(q >> 2, xc, rank);
      const unsigned so = wsrc + p * WBUF_SRC + (q & 3) * ITEM_B;
      const unsigned eo = so + 8 * 1024 + h * 512;
      bool need_flag = p == rank;
      while (__hip_atomic_load(ctl + 16 + slot, __ATOMIC_RELAXED, __HIP_MEMORY_SCOPE_WORKGROUP) < (unsigned)(NGW * (q / NSLOT)))
        __builtin_amdgcn_s_sleep(1);
      const bool eager = r >= EAGER_R;
      u32x4 cp[4];
      unsigned ev, ev2 = 0u;
      unsigned tlo, thi, meta;
      for (;;) {
        asm volatile("" ::: "memory");
        bool ok = true;
        if (need_flag && !dead) {
          const unsigned f = __builtin_amdgcn_raw_buffer_load_b32(
              rl, WFLAG + (p == rank ? 0 : h * FL2 * 4) + nt * 128 + p * 16 + (q & 3) * 4, 0, 17);
          ok = __builtin_amdgcn_readfirstlane(f) >= call;
        }
        if (ok) {
          ev = __builtin_amdgcn_raw_buffer_load_b32(rl, eo + lane * 4, 0, 17);
          if (eager) {
#pragma unroll
            for (int t = 0; t < 3; ++t)
              cp[t] = __builtin_amdgcn_raw_buffer_load_b128(rl, so + (4 * h + t) * 1024 + lane * 16, 0, 17);
          }
          tlo = __builtin_amdgcn_readlane(ev, 0); thi = __builtin_amdgcn_readlane(ev, 1);
          meta = __builtin_amdgcn_readlane(ev, 2);
          if (!need_flag && !dead) {
            ok = tlo != 0u && thi != 0u && meta != 0u;
            if (ok && (meta & M_FLAG)) { need_flag = true; continue; }
          }
        }
        if (ok) {
          const bool rawm = (meta & M_RAW) != 0u;
          const int cnt = rawm ? 0 : (int)(meta & 0xffffu);
          if (!eager) {
#pragma unroll
            for (int t = 0; t < 3; ++t)
              cp[t] = __builtin_amdgcn_raw_buffer_load_b128(rl, so + (4 * h + t) * 1024 + lane * 16, 0, 17);
          }
          if (rawm) cp[3] = __builtin_amdgcn_raw_buffer_load_b128(rl, so + (4 * h + 3) * 1024 + lane * 16, 0, 17);
          if (cnt > 120) ev2 = __builtin_amdgcn_raw_buffer_load_b32(rl, eo + 256 + lane * 4, 0, 17);
          if (!need_flag && !dead) {
            unsigned mn = min(min(cp[0][0], cp[0][1]), min(cp[0][2], cp[0][3]));
            mn = min(mn, min(min(cp[1][0], cp[1][1]), min(cp[1][2], cp[1][3])));
            mn = min(mn, min(min(cp[2][0], cp[2][1]), min(cp[2][2], cp[2][3])));
            if (rawm) mn = min(mn, min(min(cp[3][0], cp[3][1]), min(cp[3][2], cp[3][3])));
            if (lane * 2 - 8 < cnt) mn = min(mn, ev);
            if (lane * 2 + 120 < cnt) mn = min(mn, ev2);
            ok = !__any(mn == 0u);
          }
          if (ok) break;
        }
        if (timed_out(tmo, call, t0, spins)) {
          dead = true;
          if (lane == 0) __hip_atomic_store(ctl + 25, 1u, __ATOMIC_RELAXED, __HIP_MEMORY_SCOPE_WORKGROUP);
        } else {
          __builtin_amdgcn_s_sleep(POLL_SLEEP);
        }
      }
      lu32x4* dst = (lu32x4*)(smem + slot * 8192 + h * 1024 + lane * 16);
      if ((meta & M_RAW) == 0u) {
        tlo &= 0x60606060u; thi &= 0x60606060u;
        const u32x4 pq = cp[0] ^ WIRE_K, qq = cp[1] ^ WIRE_K, rq = cp[2] ^ WIRE_K;
        const u32x4 dd = ((pq >> 5) & 0x03030303u) | ((qq >> 3) & 0x0C0C0C0Cu) | ((rq << 1) & 0x80808080u) |
                         ((rq >> 1) & 0x10101010u);
        const u32x4 s0 = (pq >> 2) & 0x07070707u, s1 = (qq >> 2) & 0x07070707u, s2 = (rq >> 2) & 0x07070707u,
                    s3 = (dd >> 2) & 0x07070707u;
        u32x4 h0, h1, h2, h3;
#pragma unroll
        for (int d = 0; d < 4; ++d) {
          h0[d] = __builtin_amdgcn_perm(thi, tlo, s0[d]); h1[d] = __builtin_amdgcn_perm(thi, tlo, s1[d]);
          h2[d] = __builtin_amdgcn_perm(thi, tlo, s2[d]); h3[d] = __builtin_amdgcn_perm(thi, tlo, s3[d]);
        }
        dst[0] = (pq & 0x9F9F9F9Fu) | h0;
        dst[128] = (qq & 0x9F9F9F9Fu) | h1;
        dst[256] = (rq & 0x9F9F9F9Fu) | h2;
        dst[384] = dd | h3;
        const int cnt = min((int)(meta & 0xffffu), EXC_CAP);
#pragma unroll
        for (int k = 0; k < 2; ++k) {
          const unsigned en = (ev >> (16 * k)) & 0xffffu;
          if (lane >= 4 && lane * 2 - 8 + k < cnt)
            __hip_atomic_fetch_xor((lu32*)smem + (slot * 2048 + ((en >> 4) & 0x7ffu)),
                                   (en & 3u) << (((en >> 2) & 3u) * 8u + 5u), __ATOMIC_RELAXED,
                                   __HIP_MEMORY_SCOPE_WORKGROUP);
        }
        if (cnt > 120) {
#pragma unroll
          for (int k = 0; k < 2; ++k) {
            const unsigned en = (ev2 >> (16 * k)) & 0xffffu;
            if (lane * 2 + 120 + k < cnt)
              __hip_atomic_fetch_xor((lu32*)smem + (slot * 2048 + ((en >> 4) & 0x7ffu)),
                                     (en & 3u) << (((en >> 2) & 3u) * 8u + 5u), __ATOMIC_RELAXED,
                                     __HIP_MEMORY_SCOPE_WORKGROUP);
          }
        }
      } else {
#pragma unroll
        for (int t = 0; t < 4; ++t) dst[t * 128] = cp[t] ^ WIRE_K;
      }
      asm volatile("s_waitcnt lgkmcnt(0)" ::: "memory");
      if (lane == 0)
        __hip_atomic_store(ctl + slot * 2 + h, (unsigned)(q + 1), __ATOMIC_RELAXED, __HIP_MEMORY_SCOPE_WORKGROUP);
    }
    return;
  }

  const int mw = mh * NGW + wave;
  unsigned raw[16];
  {
    __amdgpu_buffer_rsrc_t rw = mk_rsrc(P.w, wave < 4 ? KS * NN : 0);
    const unsigned lsrc = (unsigned)(16 * (lane >> 4)) * NN + 4 * (lane & 15);
    const unsigned so = (unsigned)(128 * (mh * 2 + ((wave >> 1) & 1)) + 64 * (wave & 1)) * NN + 64 * nt;
#pragma unroll
    for (int j = 0; j < 16; ++j) raw[j] = __builtin_amdgcn_raw_buffer_load_b32(rw, lsrc + j * NN, so, 0);
  }
  __builtin_amdgcn_sched_barrier(0);
  GET_CALL();
  const float scale = (*P.sx) * (*P.sw);
  if (wave < 4) {
    const int it = nt * 4 + mh * 2 + (wave >> 1), h = wave & 1;
    __builtin_amdgcn_s_setprio(3);
    lu32* hist = (lu32*)(smem + LDS_HIST + wave * 4096);
    lu16* exl = (lu16*)(smem + LDS_EXC + wave * 512);
#pragma unroll
    for (int t = 0; t < 4; ++t) ((lu32x4*)hist)[t * 64 + lane] = u32x4{0u, 0u, 0u, 0u};
    __builtin_amdgcn_s_barrier();
#pragma unroll
    for (int j = 0; j < 16; ++j)
      __hip_atomic_fetch_add(hist + (((raw[j] >> (8 * (j & 3) + 2)) & 31u) * 32u + (lane & 31)), 1u, __ATOMIC_RELAXED,
                             __HIP_MEMORY_SCOPE_WORKGROUP);
    u32x4 o[4];
#pragma unroll
    for (int jj = 0; jj < 4; ++jj) {
      unsigned a0, a1, a2, a3;
      transpose4(raw[4 * jj], raw[4 * jj + 1], raw[4 * jj + 2], raw[4 * jj + 3], a0, a1, a2, a3);
      o[0][jj] = a0; o[1][jj] = a1; o[2][jj] = a2; o[3][jj] = a3;
    }
    STAMP(12);
    unsigned tlo, thi;
    {
      const int e = ((lane >> 1) & 3) * 8 + (lane >> 3);
      unsigned tot = 0;
#pragma unroll
      for (int t = 0; t < 4; ++t) {
        const u32x4 hv = ((lu32x4*)hist)[e * 8 + (lane & 1) * 4 + t];
        tot += hv[0] + hv[1] + hv[2] + hv[3];
      }
      tot += (unsigned)__builtin_amdgcn_update_dpp(0, (int)tot, 0xB1, 0xf, 0xf, false);
      unsigned key = (tot << 2) | (unsigned)((lane >> 1) & 3);
      key = max(key, (unsigned)__builtin_amdgcn_update_dpp(0, (int)key, 0x4E, 0xf, 0xf, false));
      key = max(key, (unsigned)__builtin_amdgcn_update_dpp(0, (int)key, 0x141, 0xf, 0xf, false));
      const unsigned hb = (key & 3u) << 5;
      tlo = __builtin_amdgcn_readlane(hb, 0) | (__builtin_amdgcn_readlane(hb, 8) << 8) |
            (__builtin_amdgcn_readlane(hb, 16) << 16) | (__builtin_amdgcn_readlane(hb, 24) << 24);
      thi = __builtin_amdgcn_readlane(hb, 32) | (__builtin_amdgcn_readlane(hb, 40) << 8) |
            (__builtin_amdgcn_readlane(hb, 48) << 16) | (__builtin_amdgcn_readlane(hb, 56) << 24);
    }
    STAMP(14);
    unsigned xm[4];
    unsigned mycnt = 0;
#pragma unroll
    for (int i = 0; i < 4; ++i) {
      unsigned acc2 = 0;
#pragma unroll
      for (int d = 0; d < 4; ++d) {
        const unsigned df = (o[i][d] ^ predict_hi(o[i][d], tlo, thi)) & 0x60606060u;
        acc2 |= d == 3 ? (df << 1) : (df >> (5 - 2 * d));
      }
      xm[i] = acc2;
      mycnt += (unsigned)__builtin_popcount((acc2 | (acc2 >> 1)) & 0x55555555u);
    }
    int at = 0, cnt = 0;
#pragma unroll
    for (int bit = 0; bit < 7; ++bit) {
      const unsigned long long bal = __ballot((mycnt >> bit) & 1u);
      at += (int)__builtin_amdgcn_mbcnt_hi((unsigned)(bal >> 32), __builtin_amdgcn_mbcnt_lo((unsigned)bal, 0u)) << bit;
      cnt += __builtin_popcountll(bal) << bit;
    }
    if (cnt <= EXC_CAP) {
      while (mycnt != 0u) {
        const bool s0 = xm[0] != 0u, s1 = xm[1] != 0u, s2 = xm[2] != 0u;
        const unsigned v = s0 ? xm[0] : (s1 ? xm[1] : (s2 ? xm[2] : xm[3]));
        const unsigned pi = s0 ? 0u : (s1 ? 1u : (s2 ? 2u : 3u));
        const unsigned f2 = (unsigned)__builtin_ctz(v) & ~1u;
        const unsigned bo = ((f2 >> 1) & 3u) * 4u + (f2 >> 3);
        exl[8 + at] = (unsigned short)(((2u * pi + h) * 1024u + lane * 16u + bo) * 4u + ((v >> f2) & 3u));
        const unsigned nv = v & ~(3u << f2);
        xm[0] = s0 ? nv : xm[0];
        xm[1] = (!s0 && s1) ? nv : xm[1];
        xm[2] = (!s0 && !s1 && s2) ? nv : xm[2];
        xm[3] = (!s0 && !s1 && !s2) ? nv : xm[3];
        ++at; --mycnt;
      }
    }
    STAMP(15);
    const bool rawm = cnt > EXC_CAP;
    const unsigned dsto = WBUF + par * WBUF_PAR + rank * WBUF_SRC + it * ITEM_B + lane * 16;
    if (!rawm) {
#pragma unroll
      for (int d = 0; d < 4; ++d) {
        const unsigned dd = o[3][d];
        o[0][d] = bfi(0x60606060u, dd << 5, o[0][d]);
        o[1][d] = bfi(0x60606060u, dd << 3, o[1][d]);
        o[2][d] = bfi(0x20202020u, dd << 1, bfi(0x40404040u, dd >> 1, o[2][d]));
      }
    }
    unsigned mn = 0xffffffffu;
#pragma unroll
    for (int t = 0; t < 4; ++t) {
      o[t] ^= WIRE_K;
      const unsigned mt = min(min(o[t][0], o[t][1]), min(o[t][2], o[t][3]));
      mn = (t < 3 || rawm) ? min(mn, mt) : mn;
    }
    const bool flagm = __any(mn == 0u);
#pragma unroll
    for (int t = 0; t < 4; ++t) __builtin_amdgcn_raw_buffer_store_b128(o[t], rl, dsto + (4 * h + t) * 1024, 0, 17);
    if (lane == 0)
      *(lu32x4*)exl = u32x4{tlo | 1u, thi | 1u, M_VALID | (flagm ? M_FLAG : 0u) | (rawm ? M_RAW : (unsigned)cnt), 0xffffffffu};
    asm volatile("s_waitcnt lgkmcnt(0)" ::: "memory");
    {
      const u32x4 ev = ((lu32x4*)exl)[lane & 31];
      __builtin_amdgcn_raw_buffer_store_b128(ev, rl, lane < 32 ? dsto + 8 * 1024 + h * 512 : OOB, 0, 17);
    }
    STAMP(9);
    asm volatile("s_waitcnt vmcnt(0)" ::: "memory");
    if (lane == 0) {
      if (__hip_atomic_fetch_add(ctl + 26 + (wave >> 1), 1u, __ATOMIC_RELAXED, __HIP_MEMORY_SCOPE_WORKGROUP) == 1u)
        __hip_atomic_store((gu32*)(myws + WFLAG) + (it >> 2) * 32 + rank * 4 + (it & 3), call, __ATOMIC_RELAXED,
                           __HIP_MEMORY_SCOPE_SYSTEM);
    }
    STAMP(13);
    __builtin_amdgcn_s_setprio(0);
  } else {
    __builtin_amdgcn_s_barrier();
  }
  if (wave == NGW - 1) {
    gu32* tmo = (gu32*)(myws + TMO);
    const unsigned long long t0 = __builtin_amdgcn_s_memrealtime();
    unsigned spins = 0;
    for (;;) {
      asm volatile("" ::: "memory");
      const u32x2 xf = __builtin_amdgcn_raw_buffer_load_b64(rl, XFLAG + mh * 512 + lane * 8, 0, 17);
      if (__all(min(xf[0], xf[1]) >= call)) break;
      if (timed_out(tmo, call, t0, spins)) {
        if (lane == 0) __hip_atomic_store(ctl + 25, 1u, __ATOMIC_RELAXED, __HIP_MEMORY_SCOPE_WORKGROUP);
        break;
      }
      __builtin_amdgcn_s_sleep(8);
    }
    asm volatile("s_waitcnt lgkmcnt(0)" ::: "memory");
    if (lane == 0) __hip_atomic_store(ctl + 24, 1u, __ATOMIC_RELAXED, __HIP_MEMORY_SCOPE_WORKGROUP);
  }
  while (__hip_atomic_load(ctl + 24, __ATOMIC_RELAXED, __HIP_MEMORY_SCOPE_WORKGROUP) == 0u) __builtin_amdgcn_s_sleep(4);
  asm volatile("" ::: "memory");
  STAMP(1);
  f32x4 acc[2][4];
#pragma unroll
  for (int m = 0; m < 2; ++m)
#pragma unroll
    for (int i = 0; i < 4; ++i) acc[m][i] = f32x4{0.f, 0.f, 0.f, 0.f};

  float* outp = P.out;
  asm volatile("" : "+s"(outp));
  __amdgpu_buffer_rsrc_t ro = mk_rsrc(outp, MS * NN * 4);
  const unsigned oo = ((32 * mw + 4 * (lane >> 4)) * NN + 64 * nt + 4 * (lane & 15)) * 4;
  const unsigned xbase = XBUF + par * XBUF_PAR + mw * 16384;
#pragma unroll 1
  for (int s = 0; s < NR; ++s) {
    const int p = src_order(s, xc, rank);
    u32x4 A[2][4][2];
#pragma unroll
    for (int m = 0; m < 2; ++m)
#pragma unroll
      for (int h = 0; h < 8; ++h)
        A[m][h >> 1][h & 1] =
            __builtin_amdgcn_raw_buffer_load_b128(rl, lane * 16 + (m * 8 + h) * 1024, xbase + p * XBUF_SRC, 0);
#pragma unroll
    for (int ks = 0; ks < 4; ++ks) {
      const int q = s * 4 + ks, slot = q & (NSLOT - 1);
      for (;;) {
        const unsigned f0 = __hip_atomic_load(ctl + slot * 2, __ATOMIC_RELAXED, __HIP_MEMORY_SCOPE_WORKGROUP);
        const unsigned f1 = __hip_atomic_load(ctl + slot * 2 + 1, __ATOMIC_RELAXED, __HIP_MEMORY_SCOPE_WORKGROUP);
        if (min(f0, f1) >= (unsigned)(q + 1)) break;
        __builtin_amdgcn_s_sleep(G_SLEEP);
      }
      asm volatile("" ::: "memory");
      u32x4 B[8];
      const lu32x4* bp = (const lu32x4*)(smem + slot * 8192 + lane * 16);
#pragma unroll
      for (int t = 0; t < 8; ++t) B[t] = bp[t * 64];
      asm volatile("s_waitcnt lgkmcnt(0)" ::: "memory");
      if (lane == 0) __hip_atomic_fetch_add(ctl + 16 + slot, 1u, __ATOMIC_RELAXED, __HIP_MEMORY_SCOPE_WORKGROUP);
#pragma unroll
      for (int i = 0; i < 4; ++i) {
        const v8i bb = {(int)B[2 * i][0], (int)B[2 * i][1], (int)B[2 * i][2], (int)B[2 * i][3],
                        (int)B[2 * i + 1][0], (int)B[2 * i + 1][1], (int)B[2 * i + 1][2], (int)B[2 * i + 1][3]};
#pragma unroll
        for (int m = 0; m < 2; ++m) {
          const v8i aa = {(int)A[m][ks][0][0], (int)A[m][ks][0][1], (int)A[m][ks][0][2], (int)A[m][ks][0][3],
                          (int)A[m][ks][1][0], (int)A[m][ks][1][1], (int)A[m][ks][1][2], (int)A[m][ks][1][3]};
          acc[m][i] = __builtin_amdgcn_mfma_scale_f32_16x16x128_f8f6f4(aa, bb, acc[m][i], 0, 1, 0, 0x7f7f7f7f, 0,
                                                                      0x7f7f7f7f);
        }
      }
      asm volatile("" : "+v"(acc[0][0]), "+v"(acc[0][1]), "+v"(acc[0][2]), "+v"(acc[0][3]), "+v"(acc[1][0]),
                   "+v"(acc[1][1]), "+v"(acc[1][2]), "+v"(acc[1][3]));
    }
    if (wave >= 4) STAMP(2 + s);
  }

  STAMP(10);
  const bool bad = __builtin_amdgcn_readfirstlane(
                       __hip_atomic_load(ctl + 25, __ATOMIC_RELAXED, __HIP_MEMORY_SCOPE_WORKGROUP)) != 0u;
  const float sc = bad ? __builtin_nanf("") : scale;
  const float sl = sc * -1.4426950408889634f;
  constexpr float LIM = 60.f * 1.4426950408889634f;
#pragma unroll
  for (int m = 0; m < 2; ++m) {
    f32x4 y[4], rc[4];
#pragma unroll
    for (int i = 0; i < 4; ++i) {
      y[i] = acc[m][i] * sc;
      const f32x4 t = acc[m][i] * sl;
      f32x4 e;
#pragma unroll
      for (int r = 0; r < 4; ++r) e[r] = __builtin_amdgcn_exp2f(__builtin_amdgcn_fmed3f(t[r], -LIM, LIM));
      e = e + 1.f;
#pragma unroll
      for (int r = 0; r < 4; ++r) rc[i][r] = __builtin_amdgcn_rcpf(e[r]);
    }
#pragma unroll
    for (int r = 0; r < 4; ++r) {
      float o0, o1, o2, o3;
      asm("s_nop 0\n\tv_mul_f32 %0, %4, %8\n\tv_mul_f32 %1, %5, %9\n\tv_mul_f32 %2, %6, %10\n\tv_mul_f32 %3, %7, %11"
          : "=&v"(o0), "=&v"(o1), "=&v"(o2), "=&v"(o3)
          : "v"(y[0][r]), "v"(y[1][r]), "v"(y[2][r]), "v"(y[3][r]), "v"(rc[0][r]), "v"(rc[1][r]), "v"(rc[2][r]), "v"(rc[3][r]));
      const f32x4 o = {o0, o1, o2, o3};
      __builtin_amdgcn_raw_buffer_store_b128(__builtin_bit_cast(u32x4, o), ro, oo + (16 * m + r) * NN * 4, 0, OUT_AUX);
    }
  }
  STAMP(11);
}

extern "C" void kernel_launch(int rank, int nranks, ncclComm_t comm, hipStream_t stream, void* const* d_in,
                              const int* in_sizes, int n_in, void* d_out, int out_size, const int* in_dtypes,
                              int out_dtype, void* const* d_out_all, void* const* d_ws_all, size_t ws_bytes) {
  if (nranks != NR || ws_bytes < WS_USED || (unsigned)rank >= (unsigned)NR) {
    fprintf(stderr, "unexpected configuration\n");
    return;
  }
  Params P;
  for (int p = 0; p < NR; ++p) P.ws[p] = (char*)d_ws_all[p];
  P.x = (const unsigned char*)d_in[0];
  P.w = (const unsigned char*)d_in[1];
  P.sx = (const float*)d_in[2];
  P.sw = (const float*)d_in[3];
  P.out = (float*)d_out;
  P.rank = rank;
  P.myws = P.ws[rank];
  hipLaunchKernelGGL(gemm_rs_kernel, dim3(256), dim3(768), 0, stream, P);
}
```

```cpp
#include <hip/hip_runtime.h>
#include <rccl/rccl.h>
#include <cstdio>

typedef unsigned u32x4 __attribute__((ext_vector_type(4)));
typedef unsigned u32x2 __attribute__((ext_vector_type(2)));
typedef int v8i __attribute__((ext_vector_type(8)));
typedef float f32x4 __attribute__((ext_vector_type(4)));
typedef __attribute__((address_space(1))) unsigned gu32;
typedef __attribute__((address_space(3))) unsigned lu32;
typedef __attribute__((address_space(3))) unsigned short lu16;
typedef __attribute__((address_space(3))) u32x4 lu32x4;

#define POLL_SLEEP 4
#define SCRUB_SLEEP 4
#define EAGER_R 14
#define EXC_GRAN 2
#define G_SLEEP 1
#define OUT_AUX 17
constexpr int NR = 8;
constexpr int NN = 4096;
constexpr int KS = 512;
constexpr int MS = 512;

constexpr unsigned XBUF_SRC = 256u << 10;
constexpr unsigned XBUF_PAR = NR * XBUF_SRC;
constexpr unsigned XBUF = 0;
constexpr unsigned ITEM_B = 9u << 10;
constexpr unsigned WBUF_SRC = 256u * ITEM_B;
constexpr unsigned WBUF_PAR = NR * WBUF_SRC;
constexpr unsigned WBUF = 2 * XBUF_PAR;
constexpr unsigned FLAGS = WBUF + 2 * WBUF_PAR;
constexpr unsigned WFLAG = FLAGS;
constexpr int FL2 = 2048;
constexpr unsigned XFLAG = FLAGS + (16u << 10);
constexpr unsigned CNT = FLAGS + (32u << 10);
constexpr unsigned TMO = FLAGS + (64u << 10);
constexpr unsigned WCTR = FLAGS + (65u << 10);
constexpr unsigned WS_USED = FLAGS + (128u << 10);
constexpr unsigned OOB = 0xFFFFFF00u;
constexpr int EXC_CAP = 248;
constexpr unsigned SCTR = FLAGS + (66u << 10);
#define WIRE_K_ 0xA5C3E187u
constexpr unsigned WIRE_K = WIRE_K_;
constexpr int NSCRUB = 80;
constexpr unsigned M_RAW = 1u << 16, M_FLAG = 1u << 17, M_VALID = 1u << 31;

#define NWP_ 16
constexpr int NWP = NWP_;
static_assert(NWP == 16, "role map assumes 16 w pushers and 80 scrubbers");
constexpr unsigned long long DEADLINE_TICKS = 50000000ull;

constexpr int NSLOT = 8;
constexpr int LDS_CTL = NSLOT * 8192;
constexpr int LDS_HIST = LDS_CTL + 1024;
constexpr int LDS_EXC = LDS_HIST + 16384;
constexpr int LDS_BYTES = 96 * 1024;
constexpr int NGW = 8;
constexpr int NLD = 4;

#define STAMP(i) do {} while (0)

struct Params {
  char* ws[NR];
  char* myws;
  const unsigned char* x;
  const unsigned char* w;
  const float* sx;
  const float* sw;
  float* out;
  int rank;
};

__device__ __forceinline__ __amdgpu_buffer_rsrc_t mk_rsrc(const void* p, unsigned bytes) {
  return __builtin_amdgcn_make_buffer_rsrc(const_cast<void*>(p), 0, (int)bytes, 0x00020000);
}

__device__ __forceinline__ void transpose4(unsigned r0, unsigned r1, unsigned r2, unsigned r3,
                                           unsigned& o0, unsigned& o1, unsigned& o2, unsigned& o3) {
  const unsigned t0 = __builtin_amdgcn_perm(r1, r0, 0x05010400u);
  const unsigned t1 = __builtin_amdgcn_perm(r1, r0, 0x07030602u);
  const unsigned t2 = __builtin_amdgcn_perm(r3, r2, 0x05010400u);
  const unsigned t3 = __builtin_amdgcn_perm(r3, r2, 0x07030602u);
  o0 = __builtin_amdgcn_perm(t2, t0, 0x05040100u);
  o1 = __builtin_amdgcn_perm(t2, t0, 0x07060302u);
  o2 = __builtin_amdgcn_perm(t3, t1, 0x05040100u);
  o3 = __builtin_amdgcn_perm(t3, t1, 0x07060302u);
}

__device__ __forceinline__ bool timed_out(gu32* tmo, unsigned call, unsigned long long t0, unsigned& spins) {
  if ((++spins & 63u) != 0u) return false;
  if (__hip_atomic_load(tmo, __ATOMIC_RELAXED, __HIP_MEMORY_SCOPE_AGENT) == call ||
      __builtin_amdgcn_s_memrealtime() - t0 > DEADLINE_TICKS) {
    __hip_atomic_store(tmo, call, __ATOMIC_RELAXED, __HIP_MEMORY_SCOPE_AGENT);
    return true;
  }
  return false;
}

__device__ __forceinline__ int stream_item(int it, int rank) {
  return ((((it >> 5) + rank) & 7) * 8 + (it & 7)) * 4 + ((it >> 3) & 3);
}
__device__ __forceinline__ int src_order(int s, int xc, int rank) {
  if (s == 0) return rank;
  const int own = (xc - rank) & 7;
  const int slot = (s - 1 < own) ? s - 1 : s;
  return (xc - slot) & 7;
}

__device__ __forceinline__ unsigned predict_hi(unsigned v, unsigned tlo, unsigned thi) {
  return __builtin_amdgcn_perm(thi, tlo, (v >> 2) & 0x07070707u);
}
__device__ __forceinline__ unsigned bfi(unsigned mask, unsigned a, unsigned b) { return (a & mask) | (b & ~mask); }

__global__ void __launch_bounds__(768, 3) gemm_rs_kernel(Params P) {
  __shared__ __attribute__((aligned(16))) char smem[LDS_BYTES];
  const int wave = __builtin_amdgcn_readfirstlane(threadIdx.x >> 6);
  const int lane = threadIdx.x & 63;
  const int b = blockIdx.x;
  const int rank = P.rank;
  char* const myws = P.myws;
  lu32* const ctl = (lu32*)(smem + LDS_CTL);

  const int xc = b & 7, idx = b >> 3;
  const bool service = idx < 4 || idx >= 20;
  if (service && wave != 0) return;

  STAMP(0);
  __amdgpu_buffer_rsrc_t rl = mk_rsrc(myws, WS_USED);
  const unsigned cnt_off = CNT + (b * 16 + wave) * 4;
  const unsigned cnt_prev = __builtin_amdgcn_raw_buffer_load_b32(rl, cnt_off, 0, 0);
#define GET_CALL()                                                                        \
  const unsigned call = __builtin_amdgcn_readfirstlane(cnt_prev) + 1u;                    \
  __builtin_amdgcn_raw_buffer_store_b32(call, rl, lane == 0 ? cnt_off : OOB, 0, 0);       \
  const unsigned par = call & 1u

  if (service) {
    const int c = lane & 15, g = lane >> 4;
    if (idx >= 22) {
      GET_CALL();
      const int sidx = (idx - 22) * 8 + xc;
      {
        gu32* tmo = (gu32*)(myws + TMO);
        const unsigned long long t0 = __builtin_amdgcn_s_memrealtime();
        unsigned spins = 0;
        __builtin_amdgcn_s_sleep(127);
        for (;;) {
          asm volatile("" ::: "memory");
          const unsigned f = __builtin_amdgcn_raw_buffer_load_b32(rl, XFLAG + (sidx & 31) * 32 + (lane & 7) * 4, 0, 17);
          if (__all(f >= call)) break;
          if (timed_out(tmo, call, t0, spins)) break;
          __builtin_amdgcn_s_sleep(16);
        }
      }
      const u32x4 z = {0u, 0u, 0u, 0u};
      for (int u = sidx; u < 7 * 256; u += NSCRUB) {
        const unsigned zo = WBUF + (par ^ 1u) * WBUF_PAR + ((rank + 1 + (u >> 8)) & 7) * WBUF_SRC + (u & 255) * ITEM_B + lane * 16;
#pragma unroll
        for (int t = 0; t < 9; ++t) __builtin_amdgcn_raw_buffer_store_b128(z, rl, zo + t * 1024, 0, 17);
        asm volatile("s_waitcnt vmcnt(9)" ::: "memory");
        __builtin_amdgcn_s_sleep(SCRUB_SLEEP);
      }
      asm volatile("s_waitcnt vmcnt(0)" ::: "memory");
      if (lane == 0) __hip_atomic_fetch_add((gu32*)(myws + SCTR), 1u, __ATOMIC_RELAXED, __HIP_MEMORY_SCOPE_AGENT);
      STAMP(1);
    } else if (idx >= 20) {
      const int v = (idx - 20) * 8 + xc;
      GET_CALL();
      __amdgpu_buffer_rsrc_t rs[NR - 1];
#pragma unroll
      for (int j = 0; j < NR - 1; ++j) rs[j] = mk_rsrc(P.ws[(rank + 1 + (j + v) % 7) & 7], WS_USED);
      gu32* fbase = (gu32*)(P.ws[(rank + 1 + lane) & 7] + WFLAG) + rank * 4;
      asm volatile("" : "+v"(fbase));
      gu32* ctr = (gu32*)(myws + WCTR) + par * 32;
      if (v == 0 && lane == 0) *((gu32*)(myws + WCTR) + (par ^ 1u) * 32) = 0u;
      unsigned one = 1u;
      asm volatile("" : "+v"(one));
      gu32* tmo = (gu32*)(myws + TMO);
      const unsigned long long t0 = __builtin_amdgcn_s_memrealtime();
      unsigned spins = 0;
      const unsigned lfo = WFLAG + lane * 128 + rank * 16;
      bool allset = false;
      auto wait_item = [&](int ai) {
        for (;;) {
          asm volatile("" ::: "memory");
          const unsigned f = __builtin_amdgcn_raw_buffer_load_b32(rl, WFLAG + (ai >> 2) * 128 + rank * 16 + (ai & 3) * 4, 0, 17);
          if (__builtin_amdgcn_readfirstlane(f) >= call) break;
          if (timed_out(tmo, call, t0, spins)) break;
          __builtin_amdgcn_s_sleep(1);
        }
      };
      constexpr int NU = 512;
      const unsigned ibase = WBUF + par * WBUF_PAR + rank * WBUF_SRC;
      auto load_unit = [&](int u, u32x4 (&d)[5]) {
        const int ai = stream_item(min(u, NU - 1) >> 1, rank), hh = u & 1;
        if (!allset) {
          asm volatile("" ::: "memory");
          const u32x4 lf = __builtin_amdgcn_raw_buffer_load_b128(rl, lfo, 0, 17);
          allset = __all(min(min(lf[0], lf[1]), min(lf[2], lf[3])) >= call);
          if (!allset) wait_item(ai);
        }
        const unsigned so = ibase + ai * ITEM_B + lane * 16;
#pragma unroll
        for (int t = 0; t < 4; ++t) d[t] = __builtin_amdgcn_raw_buffer_load_b128(rl, so + (4 * hh + t) * 1024, 0, 0);
        d[4] = __builtin_amdgcn_raw_buffer_load_b128(rl, lane < 32 ? so + 8 * 1024 + hh * 512 : OOB, 0, 0);
      };
      int cur = v, n1 = v + NWP, n2 = v + 2 * NWP;
      auto step = [&](u32x4 (&oc)[5], u32x4 (&on)[5], u32x4 (&ol)[5], unsigned& tp, unsigned& tn) {
        load_unit(n2, ol);
        unsigned long long sv;
        asm volatile("s_mov_b64 %1, exec\n\ts_mov_b64 exec, 1\n\tglobal_atomic_add %0, %2, %3, off sc0\n\ts_mov_b64 exec, %1"
                     : "=&v"(tn), "=&s"(sv) : "v"(ctr), "v"(one) : "memory");
        __builtin_amdgcn_sched_barrier(0);
        const int ai = stream_item(cur >> 1, rank), hh = cur & 1;
        const unsigned dsto = ibase + ai * ITEM_B + hh * 4096 + lane * 16;
        const unsigned m0 = __builtin_amdgcn_readlane(oc[4][2], 0);
#pragma unroll
        for (int t = 0; t < 3; ++t)
#pragma unroll
          for (int j = 0; j < NR - 1; ++j) __builtin_amdgcn_raw_buffer_store_b128(oc[t], rs[j], dsto + t * 1024, 0, 17);
        if (m0 & M_RAW) {
#pragma unroll
          for (int j = 0; j < NR - 1; ++j) __builtin_amdgcn_raw_buffer_store_b128(oc[3], rs[j], dsto + 3 * 1024, 0, 17);
        }
        if (cur == 448) {
          while (__hip_atomic_load((gu32*)(myws + SCTR), __ATOMIC_RELAXED, __HIP_MEMORY_SCOPE_AGENT) < NSCRUB * call) {
            if (timed_out(tmo, call, t0, spins)) break;
            __builtin_amdgcn_s_sleep(2);
          }
        }
        {
          const unsigned eo = ((unsigned)(lane / EXC_GRAN) * (16u * EXC_GRAN) < 16u + 2u * (m0 & 0xffffu))
                                  ? ibase + ai * ITEM_B + 8 * 1024 + hh * 512 + lane * 16 : OOB;
#pragma unroll
          for (int j = 0; j < NR - 1; ++j) __builtin_amdgcn_raw_buffer_store_b128(oc[4], rs[j], eo, 0, 17);
        }
        if (m0 & M_FLAG) {
          asm volatile("s_waitcnt vmcnt(0)" ::: "memory");
          if (lane < NR - 1)
            __hip_atomic_store(fbase + (FL2 * hh + (ai >> 2) * 32 + (ai & 3)), call, __ATOMIC_RELAXED, __HIP_MEMORY_SCOPE_SYSTEM);
        }
        asm volatile("s_waitcnt vmcnt(62)"
                     : "+v"(tp), "+v"(on[0]), "+v"(on[1]), "+v"(on[2]), "+v"(on[3]), "+v"(on[4]) :: "memory");
        __builtin_amdgcn_sched_barrier(0);
        cur = n1; n1 = n2;
        n2 = 4 * NWP + (int)__builtin_amdgcn_readfirstlane(tp);
      };
      u32x4 oa[5], ob[5], oc3[5];
      unsigned ta = (unsigned)(v - NWP), tb, tc;
      load_unit(cur, oa);
      STAMP(2);
      load_unit(n1, ob);
      __builtin_amdgcn_s_waitcnt(0x0F70);
      for (;;) {
        step(oa, ob, oc3, ta, tb); if (cur >= NU) break;
        step(ob, oc3, oa, tb, tc); if (cur >= NU) break;
        step(oc3, oa, ob, tc, ta); if (cur >= NU) break;
      }
      STAMP(3);
      STAMP(1);
    } else {
      const int j = idx * 8 + xc;
      __amdgpu_buffer_rsrc_t rx = mk_rsrc(P.x, 4096 * KS);
      __amdgpu_buffer_rsrc_t rs[NR];
      unsigned so[NR];
#pragma unroll
      for (int t = 0; t < NR; ++t) {
        const int q = (rank + 1 + t + j) & 7;
        rs[t] = mk_rsrc(P.ws[q], WS_USED);
        so[t] = (unsigned)(MS * q + 16 * j) * KS;
      }
      gu32* fptr = (gu32*)(P.ws[lane & 7] + XFLAG) + j * 8 + rank;
      const unsigned lsrc = (unsigned)c * KS + 16 * g;
      u32x4 d[2][16];
#pragma unroll
      for (int t = 0; t < 16; ++t)
        d[0][t] = __builtin_amdgcn_raw_buffer_load_b128(rx, lsrc + 64 * (t >> 3), so[t & 7], 0);
#pragma unroll
      for (int t = 0; t < 16; ++t)
        d[1][t] = __builtin_amdgcn_raw_buffer_load_b128(rx, lsrc + 64 * (2 + (t >> 3)), so[t & 7], 0);
      __builtin_amdgcn_sched_barrier(0);
      GET_CALL();
      const unsigned dsto = XBUF + par * XBUF_PAR + rank * XBUF_SRC + j * 8192 + lane * 16;
#pragma unroll
      for (int st = 0; st < 4; ++st) {
        if (st >= 1 && st < 3) {
#pragma unroll
          for (int t = 0; t < 16; ++t)
            d[(st + 1) & 1][t] =
                __builtin_amdgcn_raw_buffer_load_b128(rx, lsrc + 64 * (2 * st + 2 + (t >> 3)), so[t & 7], 0);
        }
        if (st == 0) asm volatile("" : "+v"(fptr));
        __builtin_amdgcn_sched_barrier(0);
#pragma unroll
        for (int t = 0; t < 16; ++t)
          __builtin_amdgcn_raw_buffer_store_b128(d[st & 1][t], rs[t & 7], dsto + (2 * st + (t >> 3)) * 1024, 0, 17);
        __builtin_amdgcn_sched_barrier(0);
      }
      asm volatile("s_waitcnt vmcnt(0)" ::: "memory");
      if (lane < NR) __hip_atomic_store(fptr, call, __ATOMIC_RELAXED, __HIP_MEMORY_SCOPE_SYSTEM);
      STAMP(1);
    }
    return;
  }

  const int gi = idx - 4;
  const int nt = xc * 8 + (gi >> 1);
  const int mh = gi & 1;

  if (wave >= NGW) {
    const int ld = wave - NGW, ip = ld >> 1, h = ld & 1;
    GET_CALL();
    if (ld == 0) {
      if (lane < 32) ctl[lane] = 0u;
      asm volatile("s_waitcnt lgkmcnt(0)" ::: "memory");
    }
    __builtin_amdgcn_s_barrier();
    __builtin_amdgcn_s_setprio(3);
    gu32* tmo = (gu32*)(myws + TMO);
    const unsigned long long t0 = __builtin_amdgcn_s_memrealtime();
    bool dead = false;
    unsigned spins = 0;
    const unsigned wsrc = WBUF + par * WBUF_PAR + nt * 4 * ITEM_B;
#pragma unroll 1
    for (int r = 0; r < 16; ++r) {
      const int q = 2 * r + ip, slot = q & (NSLOT - 1);
      const int p = src_order(q >> 2, xc, rank);
      const unsigned so = wsrc + p * WBUF_SRC + (q & 3) * ITEM_B;
      const unsigned eo = so + 8 * 1024 + h * 512;
      bool need_flag = p == rank;
      while (__hip_atomic_load(ctl + 16 + slot, __ATOMIC_RELAXED, __HIP_MEMORY_SCOPE_WORKGROUP) < (unsigned)(NGW * (q / NSLOT)))
        __builtin_amdgcn_s_sleep(1);
      const bool eager = r >= EAGER_R;
      u32x4 cp[4];
      unsigned ev, ev2 = 0u;
      unsigned tlo, thi, meta;
      for (;;) {
        asm volatile("" ::: "memory");
        bool ok = true;
        if (need_flag && !dead) {
          const unsigned f = __builtin_amdgcn_raw_buffer_load_b32(
              rl, WFLAG + (p == rank ? 0 : h * FL2 * 4) + nt * 128 + p * 16 + (q & 3) * 4, 0, 17);
          ok = __builtin_amdgcn_readfirstlane(f) >= call;
        }
        if (ok) {
          ev = __builtin_amdgcn_raw_buffer_load_b32(rl, eo + lane * 4, 0, 17);
          if (eager) {
#pragma unroll
            for (int t = 0; t < 3; ++t)
              cp[t] = __builtin_amdgcn_raw_buffer_load_b128(rl, so + (4 * h + t) * 1024 + lane * 16, 0, 17);
          }
          tlo = __builtin_amdgcn_readlane(ev, 0); thi = __builtin_amdgcn_readlane(ev, 1);
          meta = __builtin_amdgcn_readlane(ev, 2);
          if (!need_flag && !dead) {
            ok = tlo != 0u && thi != 0u && meta != 0u;
            if (ok && (meta & M_FLAG)) { need_flag = true; continue; }
          }
        }
        if (ok) {
          const bool rawm = (meta & M_RAW) != 0u;
          const int cnt = rawm ? 0 : (int)(meta & 0xffffu);
          if (!eager) {
#pragma unroll
            for (int t = 0; t < 3; ++t)
              cp[t] = __builtin_amdgcn_raw_buffer_load_b128(rl, so + (4 * h + t) * 1024 + lane * 16, 0, 17);
          }
          if (rawm) cp[3] = __builtin_amdgcn_raw_buffer_load_b128(rl, so + (4 * h + 3) * 1024 + lane * 16, 0, 17);
          if (cnt > 120) ev2 = __builtin_amdgcn_raw_buffer_load_b32(rl, eo + 256 + lane * 4, 0, 17);
          if (!need_flag && !dead) {
            unsigned mn = min(min(cp[0][0], cp[0][1]), min(cp[0][2], cp[0][3]));
            mn = min(mn, min(min(cp[1][0], cp[1][1]), min(cp[1][2], cp[1][3])));
            mn = min(mn, min(min(cp[2][0], cp[2][1]), min(cp[2][2], cp[2][3])));
            if (rawm) mn = min(mn, min(min(cp[3][0], cp[3][1]), min(cp[3][2], cp[3][3])));
            if (lane * 2 - 8 < cnt) mn = min(mn, ev);
            if (lane * 2 + 120 < cnt) mn = min(mn, ev2);
            ok = !__any(mn == 0u);
          }
          if (ok) break;
        }
        if (timed_out(tmo, call, t0, spins)) {
          dead = true;
          if (lane == 0) __hip_atomic_store(ctl + 25, 1u, __ATOMIC_RELAXED, __HIP_MEMORY_SCOPE_WORKGROUP);
        } else {
          __builtin_amdgcn_s_sleep(POLL_SLEEP);
        }
      }
      lu32x4* dst = (lu32x4*)(smem + slot * 8192 + h * 1024 + lane * 16);
      if ((meta & M_RAW) == 0u) {
        tlo &= 0x60606060u; thi &= 0x60606060u;
        const u32x4 pq = cp[0] ^ WIRE_K, qq = cp[1] ^ WIRE_K, rq = cp[2] ^ WIRE_K;
        const u32x4 dd = ((pq >> 5) & 0x03030303u) | ((qq >> 3) & 0x0C0C0C0Cu) | ((rq << 1) & 0x80808080u) |
                         ((rq >> 1) & 0x10101010u);
        const u32x4 s0 = (pq >> 2) & 0x07070707u, s1 = (qq >> 2) & 0x07070707u, s2 = (rq >> 2) & 0x07070707u,
                    s3 = (dd >> 2) & 0x07070707u;
        u32x4 h0, h1, h2, h3;
#pragma unroll
        for (int d = 0; d < 4; ++d) {
          h0[d] = __builtin_amdgcn_perm(thi, tlo, s0[d]); h1[d] = __builtin_amdgcn_perm(thi, tlo, s1[d]);
          h2[d] = __builtin_amdgcn_perm(thi, tlo, s2[d]); h3[d] = __builtin_amdgcn_perm(thi, tlo, s3[d]);
        }
        dst[0] = (pq & 0x9F9F9F9Fu) | h0;
        dst[128] = (qq & 0x9F9F9F9Fu) | h1;
        dst[256] = (rq & 0x9F9F9F9Fu) | h2;
        dst[384] = dd | h3;
        const int cnt = min((int)(meta & 0xffffu), EXC_CAP);
#pragma unroll
        for (int k = 0; k < 2; ++k) {
          const unsigned en = (ev >> (16 * k)) & 0xffffu;
          if (lane >= 4 && lane * 2 - 8 + k < cnt)
            __hip_atomic_fetch_xor((lu32*)smem + (slot * 2048 + ((en >> 4) & 0x7ffu)),
                                   (en & 3u) << (((en >> 2) & 3u) * 8u + 5u), __ATOMIC_RELAXED,
                                   __HIP_MEMORY_SCOPE_WORKGROUP);
        }
        if (cnt > 120) {
#pragma unroll
          for (int k = 0; k < 2; ++k) {
            const unsigned en = (ev2 >> (16 * k)) & 0xffffu;
            if (lane * 2 + 120 + k < cnt)
              __hip_atomic_fetch_xor((lu32*)smem + (slot * 2048 + ((en >> 4) & 0x7ffu)),
                                     (en & 3u) << (((en >> 2) & 3u) * 8u + 5u), __ATOMIC_RELAXED,
                                     __HIP_MEMORY_SCOPE_WORKGROUP);
          }
        }
      } else {
#pragma unroll
        for (int t = 0; t < 4; ++t) dst[t * 128] = cp[t] ^ WIRE_K;
      }
      asm volatile("s_waitcnt lgkmcnt(0)" ::: "memory");
      if (lane == 0)
        __hip_atomic_store(ctl + slot * 2 + h, (unsigned)(q + 1), __ATOMIC_RELAXED, __HIP_MEMORY_SCOPE_WORKGROUP);
    }
    return;
  }

  const int mw = mh * NGW + wave;
  unsigned raw[16];
  {
    __amdgpu_buffer_rsrc_t rw = mk_rsrc(P.w, wave < 4 ? KS * NN : 0);
    const unsigned lsrc = (unsigned)(16 * (lane >> 4)) * NN + 4 * (lane & 15);
    const unsigned so = (unsigned)(128 * (mh * 2 + ((wave >> 1) & 1)) + 64 * (wave & 1)) * NN + 64 * nt;
#pragma unroll
    for (int j = 0; j < 16; ++j) raw[j] = __builtin_amdgcn_raw_buffer_load_b32(rw, lsrc + j * NN, so, 0);
  }
  __builtin_amdgcn_sched_barrier(0);
  GET_CALL();
  const float scale = (*P.sx) * (*P.sw);
  if (wave < 4) {
    const int it = nt * 4 + mh * 2 + (wave >> 1), h = wave & 1;
    __builtin_amdgcn_s_setprio(3);
    lu32* hist = (lu32*)(smem + LDS_HIST + wave * 4096);
    lu16* exl = (lu16*)(smem + LDS_EXC + wave * 512);
#pragma unroll
    for (int t = 0; t < 4; ++t) ((lu32x4*)hist)[t * 64 + lane] = u32x4{0u, 0u, 0u, 0u};
    __builtin_amdgcn_s_barrier();
#pragma unroll
    for (int j = 0; j < 16; ++j)
      __hip_atomic_fetch_add(hist + (((raw[j] >> (8 * (j & 3) + 2)) & 31u) * 32u + (lane & 31)), 1u, __ATOMIC_RELAXED,
                             __HIP_MEMORY_SCOPE_WORKGROUP);
    u32x4 o[4];
#pragma unroll
    for (int jj = 0; jj < 4; ++jj) {
      unsigned a0, a1, a2, a3;
      transpose4(raw[4 * jj], raw[4 * jj + 1], raw[4 * jj + 2], raw[4 * jj + 3], a0, a1, a2, a3);
      o[0][jj] = a0; o[1][jj] = a1; o[2][jj] = a2; o[3][jj] = a3;
    }
    STAMP(12);
    unsigned tlo, thi;
    {
      const int e = ((lane >> 1) & 3) * 8 + (lane >> 3);
      unsigned tot = 0;
#pragma unroll
      for (int t = 0; t < 4; ++t) {
        const u32x4 hv = ((lu32x4*)hist)[e * 8 + (lane & 1) * 4 + t];
        tot += hv[0] + hv[1] + hv[2] + hv[3];
      }
      tot += (unsigned)__builtin_amdgcn_update_dpp(0, (int)tot, 0xB1, 0xf, 0xf, false);
      unsigned key = (tot << 2) | (unsigned)((lane >> 1) & 3);
      key = max(key, (unsigned)__builtin_amdgcn_update_dpp(0, (int)key, 0x4E, 0xf, 0xf, false));
      key = max(key, (unsigned)__builtin_amdgcn_update_dpp(0, (int)key, 0x141, 0xf, 0xf, false));
      const unsigned hb = (key & 3u) << 5;
      tlo = __builtin_amdgcn_readlane(hb, 0) | (__builtin_amdgcn_readlane(hb, 8) << 8) |
            (__builtin_amdgcn_readlane(hb, 16) << 16) | (__builtin_amdgcn_readlane(hb, 24) << 24);
      thi = __builtin_amdgcn_readlane(hb, 32) | (__builtin_amdgcn_readlane(hb, 40) << 8) |
            (__builtin_amdgcn_readlane(hb, 48) << 16) | (__builtin_amdgcn_readlane(hb, 56) << 24);
    }
    STAMP(14);
    unsigned xm[4];
    unsigned mycnt = 0;
#pragma unroll
    for (int i = 0; i < 4; ++i) {
      unsigned acc2 = 0;
#pragma unroll
      for (int d = 0; d < 4; ++d) {
        const unsigned df = (o[i][d] ^ predict_hi(o[i][d], tlo, thi)) & 0x60606060u;
        acc2 |= d == 3 ? (df << 1) : (df >> (5 - 2 * d));
      }
      xm[i] = acc2;
      mycnt += (unsigned)__builtin_popcount((acc2 | (acc2 >> 1)) & 0x55555555u);
    }
    int at = 0, cnt = 0;
#pragma unroll
    for (int bit = 0; bit < 7; ++bit) {
      const unsigned long long bal = __ballot((mycnt >> bit) & 1u);
      at += (int)__builtin_amdgcn_mbcnt_hi((unsigned)(bal >> 32), __builtin_amdgcn_mbcnt_lo((unsigned)bal, 0u)) << bit;
      cnt += __builtin_popcountll(bal) << bit;
    }
    if (cnt <= EXC_CAP) {
      while (mycnt != 0u) {
        const bool s0 = xm[0] != 0u, s1 = xm[1] != 0u, s2 = xm[2] != 0u;
        const unsigned v = s0 ? xm[0] : (s1 ? xm[1] : (s2 ? xm[2] : xm[3]));
        const unsigned pi = s0 ? 0u : (s1 ? 1u : (s2 ? 2u : 3u));
        const unsigned f2 = (unsigned)__builtin_ctz(v) & ~1u;
        const unsigned bo = ((f2 >> 1) & 3u) * 4u + (f2 >> 3);
        exl[8 + at] = (unsigned short)(((2u * pi + h) * 1024u + lane * 16u + bo) * 4u + ((v >> f2) & 3u));
        const unsigned nv = v & ~(3u << f2);
        xm[0] = s0 ? nv : xm[0];
        xm[1] = (!s0 && s1) ? nv : xm[1];
        xm[2] = (!s0 && !s1 && s2) ? nv : xm[2];
        xm[3] = (!s0 && !s1 && !s2) ? nv : xm[3];
        ++at; --mycnt;
      }
    }
    STAMP(15);
    const bool rawm = cnt > EXC_CAP;
    const unsigned dsto = WBUF + par * WBUF_PAR + rank * WBUF_SRC + it * ITEM_B + lane * 16;
    if (!rawm) {
#pragma unroll
      for (int d = 0; d < 4; ++d) {
        const unsigned dd = o[3][d];
        o[0][d] = bfi(0x60606060u, dd << 5, o[0][d]);
        o[1][d] = bfi(0x60606060u, dd << 3, o[1][d]);
        o[2][d] = bfi(0x20202020u, dd << 1, bfi(0x40404040u, dd >> 1, o[2][d]));
      }
    }
    unsigned mn = 0xffffffffu;
#pragma unroll
    for (int t = 0; t < 4; ++t) {
      o[t] ^= WIRE_K;
      const unsigned mt = min(min(o[t][0], o[t][1]), min(o[t][2], o[t][3]));
      mn = (t < 3 || rawm) ? min(mn, mt) : mn;
    }
    const bool flagm = __any(mn == 0u);
#pragma unroll
    for (int t = 0; t < 4; ++t) __builtin_amdgcn_raw_buffer_store_b128(o[t], rl, dsto + (4 * h + t) * 1024, 0, 17);
    if (lane == 0)
      *(lu32x4*)exl = u32x4{tlo | 1u, thi | 1u, M_VALID | (flagm ? M_FLAG : 0u) | (rawm ? M_RAW : (unsigned)cnt), 0xffffffffu};
    asm volatile("s_waitcnt lgkmcnt(0)" ::: "memory");
    {
      const u32x4 ev = ((lu32x4*)exl)[lane & 31];
      __builtin_amdgcn_raw_buffer_store_b128(ev, rl, lane < 32 ? dsto + 8 * 1024 + h * 512 : OOB, 0, 17);
    }
    STAMP(9);
    asm volatile("s_waitcnt vmcnt(0)" ::: "memory");
    if (lane == 0) {
      if (__hip_atomic_fetch_add(ctl + 26 + (wave >> 1), 1u, __ATOMIC_RELAXED, __HIP_MEMORY_SCOPE_WORKGROUP) == 1u)
        __hip_atomic_store((gu32*)(myws + WFLAG) + (it >> 2) * 32 + rank * 4 + (it & 3), call, __ATOMIC_RELAXED,
                           __HIP_MEMORY_SCOPE_SYSTEM);
    }
    STAMP(13);
    __builtin_amdgcn_s_setprio(0);
  } else {
    __builtin_amdgcn_s_barrier();
  }
  if (wave == NGW - 1) {
    gu32* tmo = (gu32*)(myws + TMO);
    const unsigned long long t0 = __builtin_amdgcn_s_memrealtime();
    unsigned spins = 0;
    for (;;) {
      asm volatile("" ::: "memory");
      const u32x2 xf = __builtin_amdgcn_raw_buffer_load_b64(rl, XFLAG + mh * 512 + lane * 8, 0, 17);
      if (__all(min(xf[0], xf[1]) >= call)) break;
      if (timed_out(tmo, call, t0, spins)) {
        if (lane == 0) __hip_atomic_store(ctl + 25, 1u, __ATOMIC_RELAXED, __HIP_MEMORY_SCOPE_WORKGROUP);
        break;
      }
      __builtin_amdgcn_s_sleep(8);
    }
    asm volatile("s_waitcnt lgkmcnt(0)" ::: "memory");
    if (lane == 0) __hip_atomic_store(ctl + 24, 1u, __ATOMIC_RELAXED, __HIP_MEMORY_SCOPE_WORKGROUP);
  }
  while (__hip_atomic_load(ctl + 24, __ATOMIC_RELAXED, __HIP_MEMORY_SCOPE_WORKGROUP) == 0u) __builtin_amdgcn_s_sleep(4);
  asm volatile("" ::: "memory");
  STAMP(1);
  f32x4 acc[2][4];
#pragma unroll
  for (int m = 0; m < 2; ++m)
#pragma unroll
    for (int i = 0; i < 4; ++i) acc[m][i] = f32x4{0.f, 0.f, 0.f, 0.f};

  float* outp = P.out;
  asm volatile("" : "+s"(outp));
  __amdgpu_buffer_rsrc_t ro = mk_rsrc(outp, MS * NN * 4);
  const unsigned oo = ((32 * mw + 4 * (lane >> 4)) * NN + 64 * nt + 4 * (lane & 15)) * 4;
  const unsigned xbase = XBUF + par * XBUF_PAR + mw * 16384;
#pragma unroll 1
  for (int s = 0; s < NR; ++s) {
    const int p = src_order(s, xc, rank);
    u32x4 A[2][4][2];
#pragma unroll
    for (int m = 0; m < 2; ++m)
#pragma unroll
      for (int h = 0; h < 8; ++h)
        A[m][h >> 1][h & 1] =
            __builtin_amdgcn_raw_buffer_load_b128(rl, lane * 16 + (m * 8 + h) * 1024, xbase + p * XBUF_SRC, 0);
#pragma unroll
    for (int ks = 0; ks < 4; ++ks) {
      const int q = s * 4 + ks, slot = q & (NSLOT - 1);
      for (;;) {
        const unsigned f0 = __hip_atomic_load(ctl + slot * 2, __ATOMIC_RELAXED, __HIP_MEMORY_SCOPE_WORKGROUP);
        const unsigned f1 = __hip_atomic_load(ctl + slot * 2 + 1, __ATOMIC_RELAXED, __HIP_MEMORY_SCOPE_WORKGROUP);
        if (min(f0, f1) >= (unsigned)(q + 1)) break;
        __builtin_amdgcn_s_sleep(G_SLEEP);
      }
      asm volatile("" ::: "memory");
      u32x4 B[8];
      const lu32x4* bp = (const lu32x4*)(smem + slot * 8192 + lane * 16);
#pragma unroll
      for (int t = 0; t < 8; ++t) B[t] = bp[t * 64];
      asm volatile("s_waitcnt lgkmcnt(0)" ::: "memory");
      if (lane == 0) __hip_atomic_fetch_add(ctl + 16 + slot, 1u, __ATOMIC_RELAXED, __HIP_MEMORY_SCOPE_WORKGROUP);
#pragma unroll
      for (int i = 0; i < 4; ++i) {
        const v8i bb = {(int)B[2 * i][0], (int)B[2 * i][1], (int)B[2 * i][2], (int)B[2 * i][3],
                        (int)B[2 * i + 1][0], (int)B[2 * i + 1][1], (int)B[2 * i + 1][2], (int)B[2 * i + 1][3]};
#pragma unroll
        for (int m = 0; m < 2; ++m) {
          const v8i aa = {(int)A[m][ks][0][0], (int)A[m][ks][0][1], (int)A[m][ks][0][2], (int)A[m][ks][0][3],
                          (int)A[m][ks][1][0], (int)A[m][ks][1][1], (int)A[m][ks][1][2], (int)A[m][ks][1][3]};
          acc[m][i] = __builtin_amdgcn_mfma_scale_f32_16x16x128_f8f6f4(aa, bb, acc[m][i], 0, 1, 0, 0x7f7f7f7f, 0,
                                                                      0x7f7f7f7f);
        }
      }
      asm volatile("" : "+v"(acc[0][0]), "+v"(acc[0][1]), "+v"(acc[0][2]), "+v"(acc[0][3]), "+v"(acc[1][0]),
                   "+v"(acc[1][1]), "+v"(acc[1][2]), "+v"(acc[1][3]));
    }
    if (wave >= 4) STAMP(2 + s);
  }

  STAMP(10);
  const bool bad = __builtin_amdgcn_readfirstlane(
                       __hip_atomic_load(ctl + 25, __ATOMIC_RELAXED, __HIP_MEMORY_SCOPE_WORKGROUP)) != 0u;
  const float sc = bad ? __builtin_nanf("") : scale;
  const float sl = sc * -1.4426950408889634f;
  constexpr float LIM = 60.f * 1.4426950408889634f;
#pragma unroll
  for (int m = 0; m < 2; ++m) {
    f32x4 y[4], rc[4];
#pragma unroll
    for (int i = 0; i < 4; ++i) {
      y[i] = acc[m][i] * sc;
      const f32x4 t = acc[m][i] * sl;
      f32x4 e;
#pragma unroll
      for (int r = 0; r < 4; ++r) e[r] = __builtin_amdgcn_exp2f(__builtin_amdgcn_fmed3f(t[r], -LIM, LIM));
      e = e + 1.f;
#pragma unroll
      for (int r = 0; r < 4; ++r) rc[i][r] = __builtin_amdgcn_rcpf(e[r]);
    }
#pragma unroll
    for (int r = 0; r < 4; ++r) {
      float o0, o1, o2, o3;
      asm("s_nop 0\n\tv_mul_f32 %0, %4, %8\n\tv_mul_f32 %1, %5, %9\n\tv_mul_f32 %2, %6, %10\n\tv_mul_f32 %3, %7, %11"
          : "=&v"(o0), "=&v"(o1), "=&v"(o2), "=&v"(o3)
          : "v"(y[0][r]), "v"(y[1][r]), "v"(y[2][r]), "v"(y[3][r]), "v"(rc[0][r]), "v"(rc[1][r]), "v"(rc[2][r]), "v"(rc[3][r]));
      const f32x4 o = {o0, o1, o2, o3};
      __builtin_amdgcn_raw_buffer_store_b128(__builtin_bit_cast(u32x4, o), ro, oo + (16 * m + r) * NN * 4, 0, OUT_AUX);
    }
  }
  STAMP(11);
}

extern "C" void kernel_launch(int rank, int nranks, ncclComm_t comm, hipStream_t stream, void* const* d_in,
                              const int* in_sizes, int n_in, void* d_out, int out_size, const int* in_dtypes,
                              int out_dtype, void* const* d_out_all, void* const* d_ws_all, size_t ws_bytes) {
  if (nranks != NR || ws_bytes < WS_USED || (unsigned)rank >= (unsigned)NR) {
    fprintf(stderr, "unexpected configuration\n");
    return;
  }
  Params P;
  for (int p = 0; p < NR; ++p) P.ws[p] = (char*)d_ws_all[p];
  P.x = (const unsigned char*)d_in[0];
  P.w = (const unsigned char*)d_in[1];
  P.sx = (const float*)d_in[2];
  P.sw = (const float*)d_in[3];
  P.out = (float*)d_out;
  P.rank = rank;
  P.myws = P.ws[rank];
  hipLaunchKernelGGL(gemm_rs_kernel, dim3(256), dim3(768), 0, stream, P);
}
```

```cpp
#include <hip/hip_runtime.h>
#include <rccl/rccl.h>
#include <cstdio>

typedef unsigned u32x4 __attribute__((ext_vector_type(4)));
typedef unsigned u32x2 __attribute__((ext_vector_type(2)));
typedef int v8i __attribute__((ext_vector_type(8)));
typedef float f32x4 __attribute__((ext_vector_type(4)));
typedef __attribute__((address_space(1))) unsigned gu32;
typedef __attribute__((address_space(3))) unsigned lu32;
typedef __attribute__((address_space(3))) unsigned short lu16;
typedef __attribute__((address_space(3))) u32x4 lu32x4;

#define POLL_SLEEP 4
#define SCRUB_SLEEP 4
#define EAGER_R 14
#define EXC_GRAN 2
#define G_SLEEP 1
#define OUT_AUX 17
constexpr int NR = 8;
constexpr int NN = 4096;
constexpr int KS = 512;
constexpr int MS = 512;

constexpr unsigned XBUF_SRC = 256u << 10;
constexpr unsigned XBUF_PAR = NR * XBUF_SRC;
constexpr unsigned XBUF = 0;
constexpr unsigned ITEM_B = 9u << 10;
constexpr unsigned WBUF_SRC = 256u * ITEM_B;
constexpr unsigned WBUF_PAR = NR * WBUF_SRC;
constexpr unsigned WBUF = 2 * XBUF_PAR;
constexpr unsigned FLAGS = WBUF + 2 * WBUF_PAR;
constexpr unsigned WFLAG = FLAGS;
constexpr int FL2 = 2048;
constexpr unsigned XFLAG = FLAGS + (16u << 10);
constexpr unsigned CNT = FLAGS + (32u << 10);
constexpr unsigned TMO = FLAGS + (64u << 10);
constexpr unsigned WCTR = FLAGS + (65u << 10);
constexpr unsigned WS_USED = FLAGS + (128u << 10);
constexpr unsigned OOB = 0xFFFFFF00u;
constexpr int EXC_CAP = 248;
constexpr unsigned SCTR = FLAGS + (66u << 10);
#define WIRE_K_ 0xA5C3E187u
constexpr unsigned WIRE_K = WIRE_K_;
constexpr unsigned M_RAW = 1u << 16, M_FLAG = 1u << 17, M_VALID = 1u << 31;

#define NWP_ 16
constexpr int NWP = NWP_;
constexpr int SC0 = 20 + NWP / 8;
constexpr int NSCRUB = (32 - SC0) * 8;
constexpr unsigned long long DEADLINE_TICKS = 50000000ull;

constexpr int NSLOT = 8;
constexpr int LDS_CTL = NSLOT * 8192;
constexpr int LDS_HIST = LDS_CTL + 1024;
constexpr int LDS_EXC = LDS_HIST + 16384;
constexpr int LDS_BYTES = 96 * 1024;
constexpr int NGW = 8;
constexpr int NLD = 4;

struct Params {
  char* ws[NR];
  char* myws;
  const unsigned char* x;
  const unsigned char* w;
  const float* sx;
  const float* sw;
  float* out;
  int rank;
};

__device__ __forceinline__ __amdgpu_buffer_rsrc_t mk_rsrc(const void* p, unsigned bytes) {
  return __builtin_amdgcn_make_buffer_rsrc(const_cast<void*>(p), 0, (int)bytes, 0x00020000);
}

__device__ __forceinline__ void transpose4(unsigned r0, unsigned r1, unsigned r2, unsigned r3,
                                           unsigned& o0, unsigned& o1, unsigned& o2, unsigned& o3) {
  const unsigned t0 = __builtin_amdgcn_perm(r1, r0, 0x05010400u);
  const unsigned t1 = __builtin_amdgcn_perm(r1, r0, 0x07030602u);
  const unsigned t2 = __builtin_amdgcn_perm(r3, r2, 0x05010400u);
  const unsigned t3 = __builtin_amdgcn_perm(r3, r2, 0x07030602u);
  o0 = __builtin_amdgcn_perm(t2, t0, 0x05040100u);
  o1 = __builtin_amdgcn_perm(t2, t0, 0x07060302u);
  o2 = __builtin_amdgcn_perm(t3, t1, 0x05040100u);
  o3 = __builtin_amdgcn_perm(t3, t1, 0x07060302u);
}

__device__ __forceinline__ bool timed_out(gu32* tmo, unsigned call, unsigned long long t0, unsigned& spins) {
  if ((++spins & 63u) != 0u) return false;
  if (__hip_atomic_load(tmo, __ATOMIC_RELAXED, __HIP_MEMORY_SCOPE_AGENT) == call ||
      __builtin_amdgcn_s_memrealtime() - t0 > DEADLINE_TICKS) {
    __hip_atomic_store(tmo, call, __ATOMIC_RELAXED, __HIP_MEMORY_SCOPE_AGENT);
    return true;
  }
  return false;
}

__device__ __forceinline__ int stream_item(int it, int rank) {
  return ((((it >> 5) + rank) & 7) * 8 + (it & 7)) * 4 + ((it >> 3) & 3);
}
__device__ __forceinline__ int src_order(int s, int xc, int rank) {
  if (s == 0) return rank;
  const int own = (xc - rank) & 7;
  const int slot = (s - 1 < own) ? s - 1 : s;
  return (xc - slot) & 7;
}

__device__ __forceinline__ unsigned predict_hi(unsigned v, unsigned tlo, unsigned thi) {
  return __builtin_amdgcn_perm(thi, tlo, (v >> 2) & 0x07070707u);
}
__device__ __forceinline__ unsigned bfi(unsigned mask, unsigned a, unsigned b) { return (a & mask) | (b & ~mask); }

__global__ void __launch_bounds__(768, 3) gemm_rs_kernel(Params P) {
  __shared__ __attribute__((aligned(16))) char smem[LDS_BYTES];
  const int wave = __builtin_amdgcn_readfirstlane(threadIdx.x >> 6);
  const int lane = threadIdx.x & 63;
  const int b = blockIdx.x;
  const int rank = P.rank;
  char* const myws = P.myws;
  lu32* const ctl = (lu32*)(smem + LDS_CTL);

  const int xc = b & 7, idx = b >> 3;
  const bool service = idx < 4 || idx >= 20;
  if (service && wave != 0) return;

  __amdgpu_buffer_rsrc_t rl = mk_rsrc(myws, WS_USED);
  const unsigned cnt_off = CNT + (b * 16 + wave) * 4;
  const unsigned cnt_prev = __builtin_amdgcn_raw_buffer_load_b32(rl, cnt_off, 0, 0);
#define GET_CALL()                                                                        \
  const unsigned call = __builtin_amdgcn_readfirstlane(cnt_prev) + 1u;                    \
  __builtin_amdgcn_raw_buffer_store_b32(call, rl, lane == 0 ? cnt_off : OOB, 0, 0);       \
  const unsigned par = call & 1u

  if (service) {
    const int c = lane & 15, g = lane >> 4;
    if (idx >= SC0) {
      GET_CALL();
      const int sidx = (idx - SC0) * 8 + xc;
      {
        gu32* tmo = (gu32*)(myws + TMO);
        const unsigned long long t0 = __builtin_amdgcn_s_memrealtime();
        unsigned spins = 0;
        __builtin_amdgcn_s_sleep(127);
        for (;;) {
          asm volatile("" ::: "memory");
          const unsigned f = __builtin_amdgcn_raw_buffer_load_b32(rl, XFLAG + (sidx & 31) * 32 + (lane & 7) * 4, 0, 17);
          if (__all(f >= call)) break;
          if (timed_out(tmo, call, t0, spins)) break;
          __builtin_amdgcn_s_sleep(16);
        }
      }
      const u32x4 z = {0u, 0u, 0u, 0u};
      for (int u = sidx; u < 7 * 256; u += NSCRUB) {
        const unsigned zo = WBUF + (par ^ 1u) * WBUF_PAR + ((rank + 1 + (u >> 8)) & 7) * WBUF_SRC + (u & 255) * ITEM_B + lane * 16;
#pragma unroll
        for (int t = 0; t < 9; ++t) __builtin_amdgcn_raw_buffer_store_b128(z, rl, zo + t * 1024, 0, 17);
        asm volatile("s_waitcnt vmcnt(9)" ::: "memory");
        __builtin_amdgcn_s_sleep(SCRUB_SLEEP);
      }
      asm volatile("s_waitcnt vmcnt(0)" ::: "memory");
      if (lane == 0) __hip_atomic_fetch_add((gu32*)(myws + SCTR), 1u, __ATOMIC_RELAXED, __HIP_MEMORY_SCOPE_AGENT);
    } else if (idx >= 20) {
      const int v = (idx - 20) * 8 + xc;
      GET_CALL();
      __amdgpu_buffer_rsrc_t rs[NR - 1];
#pragma unroll
      for (int j = 0; j < NR - 1; ++j) rs[j] = mk_rsrc(P.ws[(rank + 1 + (j + v) % 7) & 7], WS_USED);
      gu32* fbase = (gu32*)(P.ws[(rank + 1 + lane) & 7] + WFLAG) + rank * 4;
      asm volatile("" : "+v"(fbase));
      gu32* ctr = (gu32*)(myws + WCTR) + par * 32;
      __builtin_amdgcn_raw_buffer_store_b32(0u, rl, (v == 0 && lane == 0) ? WCTR + (par ^ 1u) * 128u : OOB, 0, 17);
      unsigned one = 1u;
      asm volatile("" : "+v"(one));
      gu32* tmo = (gu32*)(myws + TMO);
      const unsigned long long t0 = __builtin_amdgcn_s_memrealtime();
      unsigned spins = 0;
      const unsigned lfo = WFLAG + lane * 128 + rank * 16;
      bool allset = false;
      auto wait_item = [&](int ai) {
        for (;;) {
          asm volatile("" ::: "memory");
          const unsigned f = __builtin_amdgcn_raw_buffer_load_b32(rl, WFLAG + (ai >> 2) * 128 + rank * 16 + (ai & 3) * 4, 0, 17);
          if (__builtin_amdgcn_readfirstlane(f) >= call) break;
          if (timed_out(tmo, call, t0, spins)) break;
          __builtin_amdgcn_s_sleep(1);
        }
      };
      constexpr int NU = 512;
      const unsigned ibase = WBUF + par * WBUF_PAR + rank * WBUF_SRC;
      auto load_unit = [&](int u, u32x4 (&d)[5]) {
        const int ai = stream_item(min(u, NU - 1) >> 1, rank), hh = u & 1;
        if (!allset) {
          asm volatile("" ::: "memory");
          const u32x4 lf = __builtin_amdgcn_raw_buffer_load_b128(rl, lfo, 0, 17);
          allset = __all(min(min(lf[0], lf[1]), min(lf[2], lf[3])) >= call);
          if (!allset) wait_item(ai);
        }
        const unsigned so = ibase + ai * ITEM_B + lane * 16;
#pragma unroll
        for (int t = 0; t < 4; ++t) d[t] = __builtin_amdgcn_raw_buffer_load_b128(rl, so + (4 * hh + t) * 1024, 0, 0);
        d[4] = __builtin_amdgcn_raw_buffer_load_b128(rl, lane < 32 ? so + 8 * 1024 + hh * 512 : OOB, 0, 0);
      };
      int cur = v, n1 = v + NWP, n2 = v + 2 * NWP;
      auto step = [&](u32x4 (&oc)[5], u32x4 (&on)[5], u32x4 (&ol)[5], unsigned& tp, unsigned& tn) {
        load_unit(n2, ol);
        unsigned long long sv;
        asm volatile("s_mov_b64 %1, exec\n\ts_mov_b64 exec, 1\n\tglobal_atomic_add %0, %2, %3, off sc0\n\ts_mov_b64 exec, %1"
                     : "=&v"(tn), "=&s"(sv) : "v"(ctr), "v"(one) : "memory");
        __builtin_amdgcn_sched_barrier(0);
        const int ai = stream_item(cur >> 1, rank), hh = cur & 1;
        const unsigned dsto = ibase + ai * ITEM_B + hh * 4096 + lane * 16;
        const unsigned m0 = __builtin_amdgcn_readlane(oc[4][2], 0);
#pragma unroll
        for (int t = 0; t < 3; ++t)
#pragma unroll
          for (int j = 0; j < NR - 1; ++j) __builtin_amdgcn_raw_buffer_store_b128(oc[t], rs[j], dsto + t * 1024, 0, 17);
        if (m0 & M_RAW) {
#pragma unroll
          for (int j = 0; j < NR - 1; ++j) __builtin_amdgcn_raw_buffer_store_b128(oc[3], rs[j], dsto + 3 * 1024, 0, 17);
        }
        if (cur == 448) {
          while (__hip_atomic_load((gu32*)(myws + SCTR), __ATOMIC_RELAXED, __HIP_MEMORY_SCOPE_AGENT) < NSCRUB * call) {
            if (timed_out(tmo, call, t0, spins)) break;
            __builtin_amdgcn_s_sleep(2);
          }
        }
        {
          const unsigned eo = ((unsigned)(lane / EXC_GRAN) * (16u * EXC_GRAN) < 16u + 2u * (m0 & 0xffffu))
                                  ? ibase + ai * ITEM_B + 8 * 1024 + hh * 512 + lane * 16 : OOB;
#pragma unroll
          for (int j = 0; j < NR - 1; ++j) __builtin_amdgcn_raw_buffer_store_b128(oc[4], rs[j], eo, 0, 17);
        }
        if (m0 & M_FLAG) {
          asm volatile("s_waitcnt vmcnt(0)" ::: "memory");
          if (lane < NR - 1)
            __hip_atomic_store(fbase + (FL2 * hh + (ai >> 2) * 32 + (ai & 3)), call, __ATOMIC_RELAXED, __HIP_MEMORY_SCOPE_SYSTEM);
        }
        asm volatile("s_waitcnt vmcnt(62)"
                     : "+v"(tp), "+v"(on[0]), "+v"(on[1]), "+v"(on[2]), "+v"(on[3]), "+v"(on[4]) :: "memory");
        __builtin_amdgcn_sched_barrier(0);
        cur = n1; n1 = n2;
        n2 = 4 * NWP + (int)__builtin_amdgcn_readfirstlane(tp);
      };
      u32x4 oa[5], ob[5], oc3[5];
      unsigned ta = (unsigned)(v - NWP), tb, tc;
      load_unit(cur, oa);
      load_unit(n1, ob);
      __builtin_amdgcn_s_waitcnt(0x0F70);
      for (;;) {
        step(oa, ob, oc3, ta, tb); if (cur >= NU) break;
        step(ob, oc3, oa, tb, tc); if (cur >= NU) break;
        step(oc3, oa, ob, tc, ta); if (cur >= NU) break;
      }
    } else {
      const int j = idx * 8 + xc;
      __amdgpu_buffer_rsrc_t rx = mk_rsrc(P.x, 4096 * KS);
      __amdgpu_buffer_rsrc_t rs[NR];
      unsigned so[NR];
#pragma unroll
      for (int t = 0; t < NR; ++t) {
        const int q = (rank + 1 + t + j) & 7;
        rs[t] = mk_rsrc(P.ws[q], WS_USED);
        so[t] = (unsigned)(MS * q + 16 * j) * KS;
      }
      gu32* fptr = (gu32*)(P.ws[lane & 7] + XFLAG) + j * 8 + rank;
      const unsigned lsrc = (unsigned)c * KS + 16 * g;
      u32x4 d[2][16];
#pragma unroll
      for (int t = 0; t < 16; ++t)
        d[0][t] = __builtin_amdgcn_raw_buffer_load_b128(rx, lsrc + 64 * (t >> 3), so[t & 7], 0);
#pragma unroll
      for (int t = 0; t < 16; ++t)
        d[1][t] = __builtin_amdgcn_raw_buffer_load_b128(rx, lsrc + 64 * (2 + (t >> 3)), so[t & 7], 0);
      __builtin_amdgcn_sched_barrier(0);
      GET_CALL();
      const unsigned dsto = XBUF + par * XBUF_PAR + rank * XBUF_SRC + j * 8192 + lane * 16;
#pragma unroll
      for (int st = 0; st < 4; ++st) {
        if (st >= 1 && st < 3) {
#pragma unroll
          for (int t = 0; t < 16; ++t)
            d[(st + 1) & 1][t] =
                __builtin_amdgcn_raw_buffer_load_b128(rx, lsrc + 64 * (2 * st + 2 + (t >> 3)), so[t & 7], 0);
        }
        if (st == 0) asm volatile("" : "+v"(fptr));
        __builtin_amdgcn_sched_barrier(0);
#pragma unroll
        for (int t = 0; t < 16; ++t)
          __builtin_amdgcn_raw_buffer_store_b128(d[st & 1][t], rs[t & 7], dsto + (2 * st + (t >> 3)) * 1024, 0, 17);
        __builtin_amdgcn_sched_barrier(0);
      }
      asm volatile("s_waitcnt vmcnt(0)" ::: "memory");
      if (lane < NR) __hip_atomic_store(fptr, call, __ATOMIC_RELAXED, __HIP_MEMORY_SCOPE_SYSTEM);
    }
    return;
  }

  const int gi = idx - 4;
  const int nt = xc * 8 + (gi >> 1);
  const int mh = gi & 1;

  if (wave >= NGW) {
    const int ld = wave - NGW, ip = ld >> 1, h = ld & 1;
    GET_CALL();
    if (ld == 0) {
      if (lane < 32) ctl[lane] = 0u;
      asm volatile("s_waitcnt lgkmcnt(0)" ::: "memory");
    }
    __builtin_amdgcn_s_barrier();
    __builtin_amdgcn_s_setprio(3);
    gu32* tmo = (gu32*)(myws + TMO);
    const unsigned long long t0 = __builtin_amdgcn_s_memrealtime();
    bool dead = false;
    unsigned spins = 0;
    const unsigned wsrc = WBUF + par * WBUF_PAR + nt * 4 * ITEM_B;
#pragma unroll 1
    for (int r = 0; r < 16; ++r) {
      const int q = 2 * r + ip, slot = q & (NSLOT - 1);
      const int p = src_order(q >> 2, xc, rank);
      const unsigned so = wsrc + p * WBUF_SRC + (q & 3) * ITEM_B;
      const unsigned eo = so + 8 * 1024 + h * 512;
      bool need_flag = p == rank;
      while (__hip_atomic_load(ctl + 16 + slot, __ATOMIC_RELAXED, __HIP_MEMORY_SCOPE_WORKGROUP) < (unsigned)(NGW * (q / NSLOT)))
        __builtin_amdgcn_s_sleep(1);
      const bool eager = r >= EAGER_R;
      u32x4 cp[4];
      unsigned ev, ev2 = 0u;
      unsigned tlo, thi, meta;
      for (;;) {
        asm volatile("" ::: "memory");
        bool ok = true;
        if (need_flag && !dead) {
          const unsigned f = __builtin_amdgcn_raw_buffer_load_b32(
              rl, WFLAG + (p == rank ? 0 : h * FL2 * 4) + nt * 128 + p * 16 + (q & 3) * 4, 0, 17);
          ok = __builtin_amdgcn_readfirstlane(f) >= call;
        }
        if (ok) {
          ev = __builtin_amdgcn_raw_buffer_load_b32(rl, eo + lane * 4, 0, 17);
          if (eager) {
#pragma unroll
            for (int t = 0; t < 3; ++t)
              cp[t] = __builtin_amdgcn_raw_buffer_load_b128(rl, so + (4 * h + t) * 1024 + lane * 16, 0, 17);
          }
          tlo = __builtin_amdgcn_readlane(ev, 0); thi = __builtin_amdgcn_readlane(ev, 1);
          meta = __builtin_amdgcn_readlane(ev, 2);
          if (!need_flag && !dead) {
            ok = tlo != 0u && thi != 0u && meta != 0u;
            if (ok && (meta & M_FLAG)) { need_flag = true; continue; }
          }
        }
        if (ok) {
          const bool rawm = (meta & M_RAW) != 0u;
          const int cnt = rawm ? 0 : (int)(meta & 0xffffu);
          if (!eager) {
#pragma unroll
            for (int t = 0; t < 3; ++t)
              cp[t] = __builtin_amdgcn_raw_buffer_load_b128(rl, so + (4 * h + t) * 1024 + lane * 16, 0, 17);
          }
          if (rawm) cp[3] = __builtin_amdgcn_raw_buffer_load_b128(rl, so + (4 * h + 3) * 1024 + lane * 16, 0, 17);
          if (cnt > 120) ev2 = __builtin_amdgcn_raw_buffer_load_b32(rl, eo + 256 + lane * 4, 0, 17);
          if (!need_flag && !dead) {
            unsigned mn = min(min(cp[0][0], cp[0][1]), min(cp[0][2], cp[0][3]));
            mn = min(mn, min(min(cp[1][0], cp[1][1]), min(cp[1][2], cp[1][3])));
            mn = min(mn, min(min(cp[2][0], cp[2][1]), min(cp[2][2], cp[2][3])));
            if (rawm) mn = min(mn, min(min(cp[3][0], cp[3][1]), min(cp[3][2], cp[3][3])));
            if (lane * 2 - 8 < cnt) mn = min(mn, ev);
            if (lane * 2 + 120 < cnt) mn = min(mn, ev2);
            ok = !__any(mn == 0u);
          }
          if (ok) break;
        }
        if (timed_out(tmo, call, t0, spins)) {
          dead = true;
          if (lane == 0) __hip_atomic_store(ctl + 25, 1u, __ATOMIC_RELAXED, __HIP_MEMORY_SCOPE_WORKGROUP);
        } else {
          __builtin_amdgcn_s_sleep(POLL_SLEEP);
        }
      }
      lu32x4* dst = (lu32x4*)(smem + slot * 8192 + h * 1024 + lane * 16);
      if ((meta & M_RAW) == 0u) {
        tlo &= 0x60606060u; thi &= 0x60606060u;
        const u32x4 pq = cp[0] ^ WIRE_K, qq = cp[1] ^ WIRE_K, rq = cp[2] ^ WIRE_K;
        const u32x4 dd = ((pq >> 5) & 0x03030303u) | ((qq >> 3) & 0x0C0C0C0Cu) | ((rq << 1) & 0x80808080u) |
                         ((rq >> 1) & 0x10101010u);
        const u32x4 s0 = (pq >> 2) & 0x07070707u, s1 = (qq >> 2) & 0x07070707u, s2 = (rq >> 2) & 0x07070707u,
                    s3 = (dd >> 2) & 0x07070707u;
        u32x4 h0, h1, h2, h3;
#pragma unroll
        for (int d = 0; d < 4; ++d) {
          h0[d] = __builtin_amdgcn_perm(thi, tlo, s0[d]); h1[d] = __builtin_amdgcn_perm(thi, tlo, s1[d]);
          h2[d] = __builtin_amdgcn_perm(thi, tlo, s2[d]); h3[d] = __builtin_amdgcn_perm(thi, tlo, s3[d]);
        }
        dst[0] = (pq & 0x9F9F9F9Fu) | h0;
        dst[128] = (qq & 0x9F9F9F9Fu) | h1;
        dst[256] = (rq & 0x9F9F9F9Fu) | h2;
        dst[384] = dd | h3;
        const int cnt = min((int)(meta & 0xffffu), EXC_CAP);
#pragma unroll
        for (int k = 0; k < 2; ++k) {
          const unsigned en = (ev >> (16 * k)) & 0xffffu;
          if (lane >= 4 && lane * 2 - 8 + k < cnt)
            __hip_atomic_fetch_xor((lu32*)smem + (slot * 2048 + ((en >> 4) & 0x7ffu)),
                                   (en & 3u) << (((en >> 2) & 3u) * 8u + 5u), __ATOMIC_RELAXED,
                                   __HIP_MEMORY_SCOPE_WORKGROUP);
        }
        if (cnt > 120) {
#pragma unroll
          for (int k = 0; k < 2; ++k) {
            const unsigned en = (ev2 >> (16 * k)) & 0xffffu;
            if (lane * 2 + 120 + k < cnt)
              __hip_atomic_fetch_xor((lu32*)smem + (slot * 2048 + ((en >> 4) & 0x7ffu)),
                                     (en & 3u) << (((en >> 2) & 3u) * 8u + 5u), __ATOMIC_RELAXED,
                                     __HIP_MEMORY_SCOPE_WORKGROUP);
          }
        }
      } else {
#pragma unroll
        for (int t = 0; t < 4; ++t) dst[t * 128] = cp[t] ^ WIRE_K;
      }
      asm volatile("s_waitcnt lgkmcnt(0)" ::: "memory");
      if (lane == 0)
        __hip_atomic_store(ctl + slot * 2 + h, (unsigned)(q + 1), __ATOMIC_RELAXED, __HIP_MEMORY_SCOPE_WORKGROUP);
    }
    return;
  }

  const int mw = mh * NGW + wave;
  unsigned raw[16];
  {
    __amdgpu_buffer_rsrc_t rw = mk_rsrc(P.w, wave < 4 ? KS * NN : 0);
    const unsigned lsrc = (unsigned)(16 * (lane >> 4)) * NN + 4 * (lane & 15);
    const unsigned so = (unsigned)(128 * (mh * 2 + ((wave >> 1) & 1)) + 64 * (wave & 1)) * NN + 64 * nt;
#pragma unroll
    for (int j = 0; j < 16; ++j) raw[j] = __builtin_amdgcn_raw_buffer_load_b32(rw, lsrc + j * NN, so, 0);
  }
  __builtin_amdgcn_sched_barrier(0);
  GET_CALL();
  const float scale = (*P.sx) * (*P.sw);
  if (wave < 4) {
    const int it = nt * 4 + mh * 2 + (wave >> 1), h = wave & 1;
    __builtin_amdgcn_s_setprio(3);
    lu32* hist = (lu32*)(smem + LDS_HIST + wave * 4096);
    lu16* exl = (lu16*)(smem + LDS_EXC + wave * 512);
#pragma unroll
    for (int t = 0; t < 4; ++t) ((lu32x4*)hist)[t * 64 + lane] = u32x4{0u, 0u, 0u, 0u};
    __builtin_amdgcn_s_barrier();
#pragma unroll
    for (int j = 0; j < 16; ++j)
      __hip_atomic_fetch_add(hist + (((raw[j] >> (8 * (j & 3) + 2)) & 31u) * 32u + (lane & 31)), 1u, __ATOMIC_RELAXED,
                             __HIP_MEMORY_SCOPE_WORKGROUP);
    u32x4 o[4];
#pragma unroll
    for (int jj = 0; jj < 4; ++jj) {
      unsigned a0, a1, a2, a3;
      transpose4(raw[4 * jj], raw[4 * jj + 1], raw[4 * jj + 2], raw[4 * jj + 3], a0, a1, a2, a3);
      o[0][jj] = a0; o[1][jj] = a1; o[2][jj] = a2; o[3][jj] = a3;
    }
    unsigned tlo, thi;
    {
      const int e = ((lane >> 1) & 3) * 8 + (lane >> 3);
      unsigned tot = 0;
#pragma unroll
      for (int t = 0; t < 4; ++t) {
        const u32x4 hv = ((lu32x4*)hist)[e * 8 + (lane & 1) * 4 + t];
        tot += hv[0] + hv[1] + hv[2] + hv[3];
      }
      tot += (unsigned)__builtin_amdgcn_update_dpp(0, (int)tot, 0xB1, 0xf, 0xf, false);
      unsigned key = (tot << 2) | (unsigned)((lane >> 1) & 3);
      key = max(key, (unsigned)__builtin_amdgcn_update_dpp(0, (int)key, 0x4E, 0xf, 0xf, false));
      key = max(key, (unsigned)__builtin_amdgcn_update_dpp(0, (int)key, 0x141, 0xf, 0xf, false));
      const unsigned hb = (key & 3u) << 5;
      tlo = __builtin_amdgcn_readlane(hb, 0) | (__builtin_amdgcn_readlane(hb, 8) << 8) |
            (__builtin_amdgcn_readlane(hb, 16) << 16) | (__builtin_amdgcn_readlane(hb, 24) << 24);
      thi = __builtin_amdgcn_readlane(hb, 32) | (__builtin_amdgcn_readlane(hb, 40) << 8) |
            (__builtin_amdgcn_readlane(hb, 48) << 16) | (__builtin_amdgcn_readlane(hb, 56) << 24);
    }
    unsigned xm[4];
    unsigned mycnt = 0;
#pragma unroll
    for (int i = 0; i < 4; ++i) {
      unsigned acc2 = 0;
#pragma unroll
      for (int d = 0; d < 4; ++d) {
        const unsigned df = (o[i][d] ^ predict_hi(o[i][d], tlo, thi)) & 0x60606060u;
        acc2 |= d == 3 ? (df << 1) : (df >> (5 - 2 * d));
      }
      xm[i] = acc2;
      mycnt += (unsigned)__builtin_popcount((acc2 | (acc2 >> 1)) & 0x55555555u);
    }
    int at = 0, cnt = 0;
#pragma unroll
    for (int bit = 0; bit < 7; ++bit) {
      const unsigned long long bal = __ballot((mycnt >> bit) & 1u);
      at += (int)__builtin_amdgcn_mbcnt_hi((unsigned)(bal >> 32), __builtin_amdgcn_mbcnt_lo((unsigned)bal, 0u)) << bit;
      cnt += __builtin_popcountll(bal) << bit;
    }
    if (cnt <= EXC_CAP) {
      while (mycnt != 0u) {
        const bool s0 = xm[0] != 0u, s1 = xm[1] != 0u, s2 = xm[2] != 0u;
        const unsigned v = s0 ? xm[0] : (s1 ? xm[1] : (s2 ? xm[2] : xm[3]));
        const unsigned pi = s0 ? 0u : (s1 ? 1u : (s2 ? 2u : 3u));
        const unsigned f2 = (unsigned)__builtin_ctz(v) & ~1u;
        const unsigned bo = ((f2 >> 1) & 3u) * 4u + (f2 >> 3);
        exl[8 + at] = (unsigned short)(((2u * pi + h) * 1024u + lane * 16u + bo) * 4u + ((v >> f2) & 3u));
        const unsigned nv = v & ~(3u << f2);
        xm[0] = s0 ? nv : xm[0];
        xm[1] = (!s0 && s1) ? nv : xm[1];
        xm[2] = (!s0 && !s1 && s2) ? nv : xm[2];
        xm[3] = (!s0 && !s1 && !s2) ? nv : xm[3];
        ++at; --mycnt;
      }
    }
    const bool rawm = cnt > EXC_CAP;
    const unsigned dsto = WBUF + par * WBUF_PAR + rank * WBUF_SRC + it * ITEM_B + lane * 16;
    if (!rawm) {
#pragma unroll
      for (int d = 0; d < 4; ++d) {
        const unsigned dd = o[3][d];
        o[0][d] = bfi(0x60606060u, dd << 5, o[0][d]);
        o[1][d] = bfi(0x60606060u, dd << 3, o[1][d]);
        o[2][d] = bfi(0x20202020u, dd << 1, bfi(0x40404040u, dd >> 1, o[2][d]));
      }
    }
    unsigned mn = 0xffffffffu;
#pragma unroll
    for (int t = 0; t < 4; ++t) {
      o[t] ^= WIRE_K;
      const unsigned mt = min(min(o[t][0], o[t][1]), min(o[t][2], o[t][3]));
      mn = (t < 3 || rawm) ? min(mn, mt) : mn;
    }
    const bool flagm = __any(mn == 0u);
#pragma unroll
    for (int t = 0; t < 4; ++t) __builtin_amdgcn_raw_buffer_store_b128(o[t], rl, dsto + (4 * h + t) * 1024, 0, 17);
    if (lane == 0)
      *(lu32x4*)exl = u32x4{tlo | 1u, thi | 1u, M_VALID | (flagm ? M_FLAG : 0u) | (rawm ? M_RAW : (unsigned)cnt), 0xffffffffu};
    asm volatile("s_waitcnt lgkmcnt(0)" ::: "memory");
    {
      const u32x4 ev = ((lu32x4*)exl)[lane & 31];
      __builtin_amdgcn_raw_buffer_store_b128(ev, rl, lane < 32 ? dsto + 8 * 1024 + h * 512 : OOB, 0, 17);
    }
    asm volatile("s_waitcnt vmcnt(0)" ::: "memory");
    if (lane == 0) {
      if (__hip_atomic_fetch_add(ctl + 26 + (wave >> 1), 1u, __ATOMIC_RELAXED, __HIP_MEMORY_SCOPE_WORKGROUP) == 1u)
        __hip_atomic_store((gu32*)(myws + WFLAG) + (it >> 2) * 32 + rank * 4 + (it & 3), call, __ATOMIC_RELAXED,
                           __HIP_MEMORY_SCOPE_SYSTEM);
    }
    __builtin_amdgcn_s_setprio(0);
  } else {
    __builtin_amdgcn_s_barrier();
  }
  if (wave == NGW - 1) {
    gu32* tmo = (gu32*)(myws + TMO);
    const unsigned long long t0 = __builtin_amdgcn_s_memrealtime();
    unsigned spins = 0;
    for (;;) {
      asm volatile("" ::: "memory");
      const u32x2 xf = __builtin_amdgcn_raw_buffer_load_b64(rl, XFLAG + mh * 512 + lane * 8, 0, 17);
      if (__all(min(xf[0], xf[1]) >= call)) break;
      if (timed_out(tmo, call, t0, spins)) {
        if (lane == 0) __hip_atomic_store(ctl + 25, 1u, __ATOMIC_RELAXED, __HIP_MEMORY_SCOPE_WORKGROUP);
        break;
      }
      __builtin_amdgcn_s_sleep(8);
    }
    asm volatile("s_waitcnt lgkmcnt(0)" ::: "memory");
    if (lane == 0) __hip_atomic_store(ctl + 24, 1u, __ATOMIC_RELAXED, __HIP_MEMORY_SCOPE_WORKGROUP);
  }
  while (__hip_atomic_load(ctl + 24, __ATOMIC_RELAXED, __HIP_MEMORY_SCOPE_WORKGROUP) == 0u) __builtin_amdgcn_s_sleep(4);
  asm volatile("" ::: "memory");
  f32x4 acc[2][4];
#pragma unroll
  for (int m = 0; m < 2; ++m)
#pragma unroll
    for (int i = 0; i < 4; ++i) acc[m][i] = f32x4{0.f, 0.f, 0.f, 0.f};

  float* outp = P.out;
  asm volatile("" : "+s"(outp));
  __amdgpu_buffer_rsrc_t ro = mk_rsrc(outp, MS * NN * 4);
  const unsigned oo = ((32 * mw + 4 * (lane >> 4)) * NN + 64 * nt + 4 * (lane & 15)) * 4;
  const unsigned xbase = XBUF + par * XBUF_PAR + mw * 16384;
#pragma unroll 1
  for (int s = 0; s < NR; ++s) {
    const int p = src_order(s, xc, rank);
    u32x4 A[2][4][2];
#pragma unroll
    for (int m = 0; m < 2; ++m)
#pragma unroll
      for (int h = 0; h < 8; ++h)
        A[m][h >> 1][h & 1] =
            __builtin_amdgcn_raw_buffer_load_b128(rl, lane * 16 + (m * 8 + h) * 1024, xbase + p * XBUF_SRC, 0);
#pragma unroll
    for (int ks = 0; ks < 4; ++ks) {
      const int q = s * 4 + ks, slot = q & (NSLOT - 1);
      for (;;) {
        const unsigned f0 = __hip_atomic_load(ctl + slot * 2, __ATOMIC_RELAXED, __HIP_MEMORY_SCOPE_WORKGROUP);
        const unsigned f1 = __hip_atomic_load(ctl + slot * 2 + 1, __ATOMIC_RELAXED, __HIP_MEMORY_SCOPE_WORKGROUP);
        if (min(f0, f1) >= (unsigned)(q + 1)) break;
        __builtin_amdgcn_s_sleep(G_SLEEP);
      }
      asm volatile("" ::: "memory");
      u32x4 B[8];
      const lu32x4* bp = (const lu32x4*)(smem + slot * 8192 + lane * 16);
#pragma unroll
      for (int t = 0; t < 8; ++t) B[t] = bp[t * 64];
      asm volatile("s_waitcnt lgkmcnt(0)" ::: "memory");
      if (lane == 0) __hip_atomic_fetch_add(ctl + 16 + slot, 1u, __ATOMIC_RELAXED, __HIP_MEMORY_SCOPE_WORKGROUP);
#pragma unroll
      for (int i = 0; i < 4; ++i) {
        const v8i bb = {(int)B[2 * i][0], (int)B[2 * i][1], (int)B[2 * i][2], (int)B[2 * i][3],
                        (int)B[2 * i + 1][0], (int)B[2 * i + 1][1], (int)B[2 * i + 1][2], (int)B[2 * i + 1][3]};
#pragma unroll
        for (int m = 0; m < 2; ++m) {
          const v8i aa = {(int)A[m][ks][0][0], (int)A[m][ks][0][1], (int)A[m][ks][0][2], (int)A[m][ks][0][3],
                          (int)A[m][ks][1][0], (int)A[m][ks][1][1], (int)A[m][ks][1][2], (int)A[m][ks][1][3]};
          acc[m][i] = __builtin_amdgcn_mfma_scale_f32_16x16x128_f8f6f4(aa, bb, acc[m][i], 0, 1, 0, 0x7f7f7f7f, 0,
                                                                      0x7f7f7f7f);
        }
      }
      asm volatile("" : "+v"(acc[0][0]), "+v"(acc[0][1]), "+v"(acc[0][2]), "+v"(acc[0][3]), "+v"(acc[1][0]),
                   "+v"(acc[1][1]), "+v"(acc[1][2]), "+v"(acc[1][3]));
    }
  }

  const bool bad = __builtin_amdgcn_readfirstlane(
                       __hip_atomic_load(ctl + 25, __ATOMIC_RELAXED, __HIP_MEMORY_SCOPE_WORKGROUP)) != 0u;
  const float sc = bad ? __builtin_nanf("") : scale;
  const float sl = sc * -1.4426950408889634f;
  constexpr float LIM = 60.f * 1.4426950408889634f;
#pragma unroll
  for (int m = 0; m < 2; ++m) {
    f32x4 y[4], rc[4];
#pragma unroll
    for (int i = 0; i < 4; ++i) {
      y[i] = acc[m][i] * sc;
      const f32x4 t = acc[m][i] * sl;
      f32x4 e;
#pragma unroll
      for (int r = 0; r < 4; ++r) e[r] = __builtin_amdgcn_exp2f(__builtin_amdgcn_fmed3f(t[r], -LIM, LIM));
      e = e + 1.f;
#pragma unroll
      for (int r = 0; r < 4; ++r) rc[i][r] = __builtin_amdgcn_rcpf(e[r]);
    }
#pragma unroll
    for (int r = 0; r < 4; ++r) {
      float o0, o1, o2, o3;
      asm("s_nop 0\n\tv_mul_f32 %0, %4, %8\n\tv_mul_f32 %1, %5, %9\n\tv_mul_f32 %2, %6, %10\n\tv_mul_f32 %3, %7, %11"
          : "=&v"(o0), "=&v"(o1), "=&v"(o2), "=&v"(o3)
          : "v"(y[0][r]), "v"(y[1][r]), "v"(y[2][r]), "v"(y[3][r]), "v"(rc[0][r]), "v"(rc[1][r]), "v"(rc[2][r]), "v"(rc[3][r]));
      const f32x4 o = {o0, o1, o2, o3};
      __builtin_amdgcn_raw_buffer_store_b128(__builtin_bit_cast(u32x4, o), ro, oo + (16 * m + r) * NN * 4, 0, OUT_AUX);
    }
  }
}

extern "C" void kernel_launch(int rank, int nranks, ncclComm_t comm, hipStream_t stream, void* const* d_in,
                              const int* in_sizes, int n_in, void* d_out, int out_size, const int* in_dtypes,
                              int out_dtype, void* const* d_out_all, void* const* d_ws_all, size_t ws_bytes) {
  if (nranks != NR || ws_bytes < WS_USED || (unsigned)rank >= (unsigned)NR) {
    fprintf(stderr, "unexpected configuration\n");
    return;
  }
  Params P;
  for (int p = 0; p < NR; ++p) P.ws[p] = (char*)d_ws_all[p];
  P.x = (const unsigned char*)d_in[0];
  P.w = (const unsigned char*)d_in[1];
  P.sx = (const float*)d_in[2];
  P.sw = (const float*)d_in[3];
  P.out = (float*)d_out;
  P.rank = rank;
  P.myws = P.ws[rank];
  hipLaunchKernelGGL(gemm_rs_kernel, dim3(256), dim3(768), 0, stream, P);
}
```
